```python
import numpy as np

kernel_name = "gpu_gemm_a2a_m4096_k4096_n16384_e5m2e5m2_dqrelu"


ABSMAX_RTOL = 0.05

COMPUTE_DTYPE = "float32"
EPILOGUE = "dequant_relu"
IN_QUANT = None
OUT_QUANT = None
IN_FP8 = "e5m2e5m2"
IN_DTYPES = {"x": "e5m2", "w_mat": "e5m2",
             "scale_x": "float32", "scale_w": "float32"}
WORLD_SIZE = 8

def _snap_fp8(x, mant_bits, max_abs, min_normal):
    a = np.abs(x.astype(np.float64))
    _m, _e = np.frexp(a)
    q = np.where(a >= min_normal, np.ldexp(1.0, _e - 1 - mant_bits),
                 min_normal * 2.0 ** -mant_bits)
    snapped = np.minimum(np.round(a / q) * q, max_abs)
    return (np.sign(x) * snapped).astype(np.float32)

def setup_inputs():

    x = _snap_fp8(np.random.randn(4096, 4096).astype(np.float32), 2, 57344.0, 2.0 ** -14)
    w_mat = _snap_fp8(np.random.randn(4096, 16384).astype(np.float32), 2, 57344.0, 2.0 ** -14)
    scale_x = (0.02 * (1.0 + np.random.rand(1))).astype(np.float32)
    scale_w = (0.02 * (1.0 + np.random.rand(1))).astype(np.float32)
    return {"x": x, "w_mat": w_mat, "scale_x": scale_x, "scale_w": scale_w}

def reference(x, w_mat, scale_x, scale_w):
    acc = (x @ w_mat).astype(np.float32)
    y = (acc * (scale_x * scale_w)).astype(np.float32)
    return np.maximum(y, 0.0).astype(np.float32)
```

```cpp
#include <hip/hip_runtime.h>
#include <rccl/rccl.h>
#include <cstdio>
#include <cstdint>

#define HIP_CHECK(expr) do { hipError_t _e = (expr); if (_e != hipSuccess) { \
    fprintf(stderr, "HIP error %s at %s:%d\n", hipGetErrorString(_e), __FILE__, __LINE__); \
    return; } } while (0)

namespace {

typedef int      v8i  __attribute__((ext_vector_type(8)));
typedef int      v4i  __attribute__((ext_vector_type(4)));
typedef int      v2i  __attribute__((ext_vector_type(2)));
typedef float    v4f  __attribute__((ext_vector_type(4)));
typedef unsigned u32x4 __attribute__((ext_vector_type(4)));
#define LDS_AS __attribute__((address_space(3)))
#define GLB_AS __attribute__((address_space(1)))

constexpr int NR = 8;
constexpr int KD = 4096;
constexpr int N_ALL = 16384;
constexpr int N_LOC = 2048;
constexpr int M_SH = 512;
constexpr int NWG = 256;
constexpr int NTHR = 512;
constexpr int NKS = 32;
constexpr int NSTEP = 64;
constexpr int TILE_B = 16384;
constexpr int NSA = 6;
constexpr int NSB = 3;
constexpr int LDS_X = 0;
constexpr int LDS_W = NSA * TILE_B;
constexpr int LDS_CTL = LDS_W + NSB * TILE_B;
constexpr int LDS_BYTES = LDS_CTL + 64;

constexpr size_t WS_CNT  = 0;
constexpr size_t WS_TMO  = WS_CNT + NWG * 128;
constexpr size_t WS_FLAG = WS_TMO + 4096;
constexpr size_t WS_X    = 65536;
constexpr size_t SLOT_B  = (size_t)M_SH * KD;
constexpr size_t WS_NEED = WS_X + 2 * SLOT_B;
static_assert(WS_FLAG + NR * NWG * 4 <= WS_X, "ws layout");

constexpr unsigned long long DEADLINE_TICKS = 30000000ull;

struct Params {
  const uint8_t* x; const uint8_t* w; const float* sx; const float* sw; float* out;
  uint8_t* ws[NR];
  int rank; int pad;
};

__device__ __forceinline__ void store_wt16(void* p, u32x4 v) {
  asm volatile("global_store_dwordx4 %0, %1, off sc0 sc1\n\ts_nop 1" :: "v"(p), "v"(v) : "memory");
}
__device__ __forceinline__ void store_out16(void* p, v4f v) {
  asm volatile("global_store_dwordx4 %0, %1, off sc0 sc1\n\ts_nop 1" :: "v"(p), "v"(v) : "memory");
}
__device__ __forceinline__ void glds16(const uint8_t* g, LDS_AS uint8_t* l) {
  __builtin_amdgcn_global_load_lds((const GLB_AS void*)g, (LDS_AS void*)l, 16, 0, 0);
}
#define WAITV(n) asm volatile("s_waitcnt vmcnt(%0)" :: "n"(n) : "memory")
#define BARRIER() asm volatile("s_waitcnt lgkmcnt(0)\n\ts_barrier" ::: "memory")

template <int PER> __device__ __forceinline__ void wait_leave(int stages) {
  switch (stages) {
    case 0: WAITV(0); break;
    case 1: WAITV(PER); break;
    case 2: WAITV(2 * PER); break;
    case 3: WAITV(3 * PER); break;
    case 4: WAITV(4 * PER); break;
    default: WAITV(5 * PER); break;
  }
}

__global__ __launch_bounds__(NTHR, 2) void gemm_ag_kernel(Params p) {
  extern __shared__ __attribute__((aligned(16))) uint8_t smem_[];
  LDS_AS uint8_t* smem = (LDS_AS uint8_t*)smem_;
  const int tid = threadIdx.x, lane = tid & 63;
  const int wid = __builtin_amdgcn_readfirstlane(tid >> 6);
  const int b = blockIdx.x;
  uint8_t* const ws = p.ws[p.rank];

  unsigned* cnt = (unsigned*)(ws + WS_CNT + (size_t)b * 128);
  const unsigned call = __builtin_amdgcn_readfirstlane(__hip_atomic_load(cnt, __ATOMIC_RELAXED, __HIP_MEMORY_SCOPE_AGENT)) + 1u;
  const size_t slot = WS_X + (size_t)(call & 1u) * SLOT_B;

  {
    const int pid = b * 8 + wid;
    const int half = pid & 1, mf = (pid >> 1) & 7, ks = (pid >> 4) & 31, blk = pid >> 9;
    const uint8_t* g = p.x + (size_t)(blk * 128 + mf * 16 + (lane & 15)) * KD + ks * 128 + half * 64 + (lane >> 4) * 16;
    const u32x4 v = *(const u32x4*)g;
    store_wt16(ws + slot + (size_t)pid * 1024 + lane * 16, v);
    WAITV(0);
    BARRIER();
    if (wid == 0) {
      if (lane < NR) __hip_atomic_store((unsigned*)(p.ws[lane] + WS_FLAG) + p.rank * NWG + b, call, __ATOMIC_RELAXED, __HIP_MEMORY_SCOPE_SYSTEM);
      if (lane == 0) __hip_atomic_store(cnt, call, __ATOMIC_RELAXED, __HIP_MEMORY_SCOPE_AGENT);
    }
  }

  const int grp = b & 7, idx = b >> 3;
  const int src = (p.rank + grp) & (NR - 1);
  const int rb = idx >> 4, cb = idx & 15;
  const uint8_t* const xsrc = p.ws[src] + slot;
  const int col0 = p.rank * N_LOC + cb * 128;

  auto kstep = [](int u) { return u < NKS ? u : NSTEP - 1 - u; };

  const int wj = wid & 1;
  const unsigned wlane = (unsigned)(lane >> 3) * N_ALL + (unsigned)(((lane & 7) ^ ((lane >> 4) & 3)) * 16);
  auto issue_w = [&](int u) {
    const int kk = kstep(u);
    const uint8_t* base = p.w + ((size_t)kk * 128 + wj * 64) * N_ALL + col0;
    LDS_AS uint8_t* l = smem + LDS_W + (u % NSB) * TILE_B + wj * 8192;
#pragma unroll
    for (int i = 0; i < 8; i++)
      glds16(base + (size_t)i * 8 * N_ALL + (wlane ^ (((i >> 1) & 1) * 64)), l + i * 1024);
  };
  auto issue_x = [&](int u) {
    const int blk = (u >> 5) * 2 + rb;
    const uint8_t* base = xsrc + ((size_t)blk * NKS + kstep(u)) * TILE_B + wj * 8192 + lane * 16;
    LDS_AS uint8_t* l = smem + LDS_X + (u % NSA) * TILE_B + wj * 8192;
#pragma unroll
    for (int i = 0; i < 8; i++) glds16(base + i * 1024, l + i * 1024);
  };

  if (wid >= 6) { for (int u = 0; u < NSB; u++) issue_w(u); }

  volatile LDS_AS unsigned* ctl = (volatile LDS_AS unsigned*)(smem + LDS_CTL);
  if (wid == 4) {
    const unsigned* f = (const unsigned*)(ws + WS_FLAG) + src * NWG + rb * 64 + lane;
    unsigned* tmo = (unsigned*)(ws + WS_TMO);
    const unsigned long long t0 = __builtin_amdgcn_s_memrealtime();
    while (true) {
      const unsigned f0 = __hip_atomic_load(f, __ATOMIC_RELAXED, __HIP_MEMORY_SCOPE_SYSTEM);
      const unsigned f1 = __hip_atomic_load(f + 128, __ATOMIC_RELAXED, __HIP_MEMORY_SCOPE_SYSTEM);
      if (__all(f0 >= call && f1 >= call)) break;
      if (__hip_atomic_load(tmo, __ATOMIC_RELAXED, __HIP_MEMORY_SCOPE_AGENT) == call ||
          __builtin_amdgcn_s_memrealtime() - t0 > DEADLINE_TICKS) {
        if (lane == 0) __hip_atomic_store(tmo, call, __ATOMIC_RELAXED, __HIP_MEMORY_SCOPE_AGENT);
        break;
      }
      __builtin_amdgcn_s_sleep(1);
    }
    __builtin_amdgcn_fence(__ATOMIC_ACQUIRE, "");
    WAITV(0);
    const unsigned t = __hip_atomic_load(tmo, __ATOMIC_RELAXED, __HIP_MEMORY_SCOPE_AGENT);
    if (lane == 0) ctl[0] = (t == call) ? 1u : 0u;
  }
  BARRIER();
  const bool bad = __builtin_amdgcn_readfirstlane(ctl[0]) != 0u;

  const int wm = (wid >> 1) & 1, wn = wid & 1;
  float* const outp = p.out + (size_t)(src * M_SH + rb * 128 + wm * 64 + (lane & 15)) * N_LOC + cb * 128 + wn * 64 + (lane >> 4) * 4;

  if (bad) {
    if (wid >= 6) WAITV(0);
    if (wid < 4) {
      const float n = __builtin_nanf("");
      const v4f nv = {n, n, n, n};
      for (int ps = 0; ps < 2; ps++)
        for (int mf = 0; mf < 4; mf++)
          for (int nf = 0; nf < 4; nf++) store_out16(outp + (size_t)(ps * 256 + mf * 16) * N_LOC + nf * 16, nv);
    }
    return;
  }

  if (wid >= 4) {
    if (wid < 6) {
      for (int u = 0; u < NSA; u++) issue_x(u);
      WAITV(8 * (NSA - 1));
      BARRIER();
      for (int u = 0; u < NSTEP; u++) {
        if (u >= 1 && u - 1 + NSA < NSTEP) issue_x(u - 1 + NSA);
        const int left = NSTEP - 2 - u;
        wait_leave<8>(left < NSA - 2 ? (left < 0 ? 0 : left) : NSA - 2);
        BARRIER();
      }
    } else {
      WAITV(8 * (NSB - 1));
      BARRIER();
      for (int u = 0; u < NSTEP; u++) {
        if (u >= 1 && u - 1 + NSB < NSTEP) issue_w(u - 1 + NSB);
        const int left = NSTEP - 2 - u;
        wait_leave<8>(left < NSB - 2 ? (left < 0 ? 0 : left) : NSB - 2);
        BARRIER();
      }
    }
    return;
  }

  const float scale = p.sx[0] * p.sw[0];
  const unsigned xaddr = (unsigned)(wm * 8192 + lane * 16);
  unsigned waddr[4];
  {
    const int g = lane >> 4, q = (lane & 15) >> 1;
    const int f = (q >> 1) | ((g & 1) << 2);
#pragma unroll
    for (int nf = 0; nf < 4; nf++)
      waddr[nf] = (unsigned)(LDS_W + (16 * g + q) * 128 + (((wn * 4 + nf) ^ f) << 4) + (lane & 1) * 8);
  }
  BARRIER();
  int sa = 0, sb = 0;
  for (int ps = 0; ps < 2; ps++) {
    v4f acc[4][4];
#pragma unroll
    for (int i = 0; i < 4; i++)
#pragma unroll
      for (int j = 0; j < 4; j++) acc[i][j] = (v4f){0.f, 0.f, 0.f, 0.f};
    for (int s = 0; s < NKS; s++) {
      LDS_AS uint8_t* xa = smem + xaddr + sa * TILE_B;
      const unsigned wo = sb * TILE_B;
      v8i wf[4], xf[4];
#pragma unroll
      for (int nf = 0; nf < 4; nf++) {
        LDS_AS uint8_t* wp = smem + waddr[nf] + wo;
        const v2i t0 = __builtin_amdgcn_ds_read_tr8_b64_v2i32((LDS_AS v2i*)(wp));
        const v2i t1 = __builtin_amdgcn_ds_read_tr8_b64_v2i32((LDS_AS v2i*)(wp + 1024));
        const v2i t2 = __builtin_amdgcn_ds_read_tr8_b64_v2i32((LDS_AS v2i*)(wp + 8192));
        const v2i t3 = __builtin_amdgcn_ds_read_tr8_b64_v2i32((LDS_AS v2i*)(wp + 9216));
        wf[nf] = (v8i){t0[0], t0[1], t1[0], t1[1], t2[0], t2[1], t3[0], t3[1]};
      }
#pragma unroll
      for (int mf = 0; mf < 4; mf++) {
        const v4i a = *(LDS_AS v4i*)(xa + mf * 2048);
        const v4i c = *(LDS_AS v4i*)(xa + mf * 2048 + 1024);
        xf[mf] = (v8i){a[0], a[1], a[2], a[3], c[0], c[1], c[2], c[3]};
      }
#pragma unroll
      for (int mf = 0; mf < 4; mf++)
#pragma unroll
        for (int nf = 0; nf < 4; nf++)
          acc[nf][mf] = __builtin_amdgcn_mfma_scale_f32_16x16x128_f8f6f4(wf[nf], xf[mf], acc[nf][mf], 1, 1, 0, 0, 0, 0);
      sa = (sa + 1 == NSA) ? 0 : sa + 1;
      sb = (sb + 1 == NSB) ? 0 : sb + 1;
      BARRIER();
    }
#pragma unroll
    for (int mf = 0; mf < 4; mf++)
#pragma unroll
      for (int nf = 0; nf < 4; nf++) {
        v4f y = acc[nf][mf] * scale;
        y[0] = fmaxf(y[0], 0.f); y[1] = fmaxf(y[1], 0.f); y[2] = fmaxf(y[2], 0.f); y[3] = fmaxf(y[3], 0.f);
        store_out16(outp + (size_t)(ps * 256 + mf * 16) * N_LOC + nf * 16, y);
      }
  }
}

}

extern "C" void kernel_launch(int rank, int nranks, ncclComm_t comm,
                              hipStream_t stream,
                              void* const* d_in, const int* in_sizes, int n_in,
                              void* d_out, int out_size,
                              const int* in_dtypes, int out_dtype,
                              void* const* d_out_all,
                              void* const* d_ws_all, size_t ws_bytes) {
  (void)comm; (void)in_sizes; (void)in_dtypes; (void)out_dtype; (void)d_out_all; (void)out_size;
  if (nranks != NR || n_in != 4 || ws_bytes < WS_NEED || (unsigned)rank >= (unsigned)NR) {
    fprintf(stderr, "kernel_launch: unexpected configuration\n");
    return;
  }
  static bool init = false;
  if (!init) {
    HIP_CHECK(hipFuncSetAttribute((const void*)gemm_ag_kernel, hipFuncAttributeMaxDynamicSharedMemorySize, LDS_BYTES));
    init = true;
  }
  Params p;
  p.x = (const uint8_t*)d_in[0]; p.w = (const uint8_t*)d_in[1];
  p.sx = (const float*)d_in[2]; p.sw = (const float*)d_in[3];
  p.out = (float*)d_out;
  for (int i = 0; i < NR; i++) p.ws[i] = (uint8_t*)d_ws_all[i];
  p.rank = rank; p.pad = 0;
  gemm_ag_kernel<<<NWG, NTHR, LDS_BYTES, stream>>>(p);
  HIP_CHECK(hipGetLastError());
}
```

```cpp
#include <hip/hip_runtime.h>
#include <rccl/rccl.h>
#include <cstdio>
#include <cstdint>

#define HIP_CHECK(expr) do { hipError_t _e = (expr); if (_e != hipSuccess) { \
    fprintf(stderr, "HIP error %s at %s:%d\n", hipGetErrorString(_e), __FILE__, __LINE__); \
    return; } } while (0)

namespace {

typedef int      v8i   __attribute__((ext_vector_type(8)));
typedef int      v4i   __attribute__((ext_vector_type(4)));
typedef int      v2i   __attribute__((ext_vector_type(2)));
typedef float    v4f   __attribute__((ext_vector_type(4)));
typedef unsigned u32x4 __attribute__((ext_vector_type(4)));
typedef unsigned u32x3 __attribute__((ext_vector_type(3)));
#define LDS_AS __attribute__((address_space(3)))

constexpr int NR = 8;
constexpr int KD = 4096;
constexpr int N_ALL = 16384;
constexpr int N_LOC = 2048;
constexpr int M_SH = 512;
constexpr int NWG = 256;
constexpr int NTHR = 512;
constexpr int NKS = 32;
constexpr int XT_B = 16384;
constexpr int WT_B = 32768;
constexpr int NSW = 3;
constexpr int NSX = 3;
constexpr int LDS_X = 0;
constexpr int LDS_W = NSX * XT_B;
constexpr int LDS_CTL = LDS_W + NSW * WT_B;
constexpr int LDS_BYTES = LDS_CTL + 64;

constexpr int GRP_P = 1536, GRP_E = 112, ECAP = 52;
constexpr int TREC = 8 * (GRP_P + GRP_E);
constexpr unsigned RAW_FLAG = 0x80u;
constexpr int SCR_B = 4608;

constexpr size_t WS_CNT  = 0;
constexpr size_t WS_TMO  = WS_CNT + NWG * 128;
constexpr size_t WS_LEAD = WS_TMO + 128;
constexpr size_t WS_INV  = WS_LEAD + 16 * 128;
constexpr size_t WS_FLAG = WS_INV + 16 * 128 + 3968;
constexpr size_t WS_X    = 262144;
constexpr size_t RAW_OFF = 1703936;
constexpr size_t SLOT_B  = RAW_OFF + (size_t)M_SH * KD;
constexpr size_t WS_NEED = WS_X + 2 * SLOT_B;
static_assert(WS_FLAG % 128 == 0 && WS_FLAG + NR * NWG * 64 <= WS_X, "ws layout");
static_assert(4 * NKS * TREC <= RAW_OFF, "ws layout");

constexpr unsigned long long DEADLINE_TICKS = 30000000ull;

struct Params {
  const uint8_t* x; const uint8_t* w; const float* sx; const float* sw; float* out;
  uint8_t* ws[NR];
  int rank; int pad;
};

#define STAMP(i) do {} while (0)
#define WAITV(n) asm volatile("s_waitcnt vmcnt(%0)" :: "n"(n) : "memory")
#define BARRIER() asm volatile("s_waitcnt lgkmcnt(0)\n\ts_barrier" ::: "memory")
#define RAW_BARRIER() asm volatile("s_barrier" ::: "memory")
#define SCHED() __builtin_amdgcn_sched_barrier(0)
#define XNT 0
#define XPOL ""

__device__ __forceinline__ void st16(void* p, u32x4 v) { asm volatile("global_store_dwordx4 %0, %1, off sc0 sc1\n\ts_nop 1" :: "v"(p), "v"(v) : "memory"); }
__device__ __forceinline__ void st12(void* p, u32x3 v) { asm volatile("global_store_dwordx3 %0, %1, off sc0 sc1\n\ts_nop 1" :: "v"(p), "v"(v) : "memory"); }
__device__ __forceinline__ void st4(void* p, unsigned v) { asm volatile("global_store_dword %0, %1, off sc0 sc1" :: "v"(p), "v"(v) : "memory"); }
__device__ __forceinline__ void store_out16(void* p, v4f v) { asm volatile("global_store_dwordx4 %0, %1, off sc0 sc1\n\ts_nop 1" :: "v"(p), "v"(v) : "memory"); }

__device__ __forceinline__ unsigned wave_sum(unsigned v) {
  v += __builtin_amdgcn_update_dpp(0u, v, 0x111, 0xf, 0xf, true);
  v += __builtin_amdgcn_update_dpp(0u, v, 0x112, 0xf, 0xf, true);
  v += __builtin_amdgcn_update_dpp(0u, v, 0x114, 0xf, 0xf, true);
  v += __builtin_amdgcn_update_dpp(0u, v, 0x118, 0xf, 0xf, true);
  v += __builtin_amdgcn_update_dpp(0u, v, 0x142, 0xa, 0xf, true);
  v += __builtin_amdgcn_update_dpp(0u, v, 0x143, 0xc, 0xf, true);
  return __builtin_amdgcn_readlane(v, 63);
}

__device__ __forceinline__ unsigned wave_scan(unsigned v) {
  v += __builtin_amdgcn_update_dpp(0u, v, 0x111, 0xf, 0xf, true);
  v += __builtin_amdgcn_update_dpp(0u, v, 0x112, 0xf, 0xf, true);
  v += __builtin_amdgcn_update_dpp(0u, v, 0x114, 0xf, 0xf, true);
  v += __builtin_amdgcn_update_dpp(0u, v, 0x118, 0xf, 0xf, true);
  v += __builtin_amdgcn_update_dpp(0u, v, 0x142, 0xa, 0xf, true);
  v += __builtin_amdgcn_update_dpp(0u, v, 0x143, 0xc, 0xf, true);
  return v;
}

__device__ __forceinline__ u32x4 unpack12(u32x3 o, unsigned b4) {
  u32x4 r;
  r[0] = (o[0] & 0x9F9F9F9Fu) + b4;
  r[1] = (o[1] & 0x9F9F9F9Fu) + b4;
  r[2] = (o[2] & 0x9F9F9F9Fu) + b4;
  r[3] = (((o[0] >> 5) & 0x03030303u) | ((o[1] >> 3) & 0x0C0C0C0Cu) | ((o[2] >> 1) & 0x10101010u) | ((o[2] << 1) & 0x80808080u)) + b4;
  return r;
}

__device__ __forceinline__ u32x4 make_srd(const void* base, unsigned bytes) {
  const unsigned long long a = (unsigned long long)base;
  return (u32x4){(unsigned)__builtin_amdgcn_readfirstlane((unsigned)a), (unsigned)__builtin_amdgcn_readfirstlane((unsigned)(a >> 32)) & 0xFFFFu,
                 (unsigned)__builtin_amdgcn_readfirstlane(bytes), 0x00020000u};
}

struct XRegs { u32x3 p[4]; unsigned e; };

__global__ __launch_bounds__(NTHR, 2) void gemm_ag_kernel(Params p) {
  extern __shared__ __attribute__((aligned(16))) uint8_t smem_[];
  LDS_AS uint8_t* smem = (LDS_AS uint8_t*)smem_;
  const int tid = threadIdx.x, lane = tid & 63;
  const int wid = __builtin_amdgcn_readfirstlane(tid >> 6);
  const int b = blockIdx.x;
  uint8_t* const ws = p.ws[p.rank];
  if (wid == 0) STAMP(0);

  unsigned* cnt = (unsigned*)(ws + WS_CNT + (size_t)b * 128);
  const unsigned call_raw = __hip_atomic_load(cnt, __ATOMIC_RELAXED, __HIP_MEMORY_SCOPE_AGENT);

  const int G = b * 4 + (wid >> 1), half = wid & 1;
  const uint8_t* gx = p.x + (size_t)((G >> 8) * 128 + (G & 7) * 16 + (lane & 15)) * KD + ((G >> 3) & 31) * 128 + half * 64 + (lane >> 4) * 16;
  const u32x4 d = *(const u32x4*)gx;
  LDS_AS uint8_t* sc = smem + LDS_X + (wid >> 1) * SCR_B + 2048;
  if (half == 0 && lane < 28) ((LDS_AS unsigned*)sc)[lane] = (unsigned)(lane * 8) | ((unsigned)(lane * 8 + 4) << 16);
  BARRIER();
  const unsigned call = __builtin_amdgcn_readfirstlane(call_raw) + 1u;
  const size_t slot = WS_X + (size_t)(call & 1u) * SLOT_B;
  const unsigned xcc = (unsigned)__builtin_amdgcn_s_getreg((3 << 11) | 20) & 0xFu;
  unsigned lead_old = call;
  if (wid == 5 && lane == 0) lead_old = __hip_atomic_fetch_max((unsigned*)(ws + WS_LEAD + xcc * 128), call, __ATOMIC_RELAXED, __HIP_MEMORY_SCOPE_AGENT);
  {
    const unsigned esum = wave_sum(__builtin_amdgcn_udot4((d[0] >> 2) & 0x1F1F1F1Fu, 0x01010101u,
                                   __builtin_amdgcn_udot4((d[1] >> 2) & 0x1F1F1F1Fu, 0x01010101u,
                                   __builtin_amdgcn_udot4((d[2] >> 2) & 0x1F1F1F1Fu, 0x01010101u,
                                   __builtin_amdgcn_udot4((d[3] >> 2) & 0x1F1F1F1Fu, 0x01010101u, 0u, false), false), false), false));
    const unsigned base = esum < 4280u ? 0u : ((esum - 4280u) >> 10 > 24u ? 24u : (esum - 4280u) >> 10);
    const unsigned b4 = base * 0x04040404u;
    unsigned c[4], P[4];
#pragma unroll
    for (int i = 0; i < 4; i++) {
      const unsigned t = (d[i] | 0x80808080u) - b4;
      c[i] = (d[i] & 0x80808080u) | (t & 0x1F1F1F1Fu);
      P[i] = (c[i] + b4) ^ d[i];
    }
    const u32x3 o = {c[0] | ((c[3] & 0x03030303u) << 5), c[1] | ((c[3] & 0x0C0C0C0Cu) << 3),
                     c[2] | ((c[3] & 0x10101010u) << 1) | ((c[3] & 0x80808080u) >> 1)};
    uint8_t* trec = ws + slot + (size_t)(G >> 3) * TREC;
    st12(trec + (G & 7) * GRP_P + half * 768 + lane * 12, o);
    const unsigned z0 = (P[0] + 0x7C7C7C7Cu) & 0x80808080u, z1 = (P[1] + 0x7C7C7C7Cu) & 0x80808080u;
    const unsigned z2 = (P[2] + 0x7C7C7C7Cu) & 0x80808080u, z3 = (P[3] + 0x7C7C7C7Cu) & 0x80808080u;
    unsigned mask = (__builtin_amdgcn_udot4(z0, 0x08040201u, __builtin_amdgcn_udot4(z1, 0x80402010u, 0u, false), false) >> 7) |
                    (__builtin_amdgcn_udot4(z2, 0x08040201u, __builtin_amdgcn_udot4(z3, 0x80402010u, 0u, false), false) << 1);
    const unsigned ncnt = (unsigned)__builtin_popcount(mask);
    const unsigned incl = wave_scan(ncnt);
    const unsigned ne = (unsigned)__builtin_amdgcn_readlane(incl, 63);
    const int dir = half ? -2 : 2;
    LDS_AS uint8_t* ea = sc + (half ? 110 : 8) + (int)(incl - ncnt) * dir;
    const unsigned long long p01 = ((unsigned long long)P[1] << 32) | P[0], p23 = ((unsigned long long)P[3] << 32) | P[2];
    const unsigned pos0 = ((unsigned)half << 10) | ((unsigned)lane << 4);
    while (mask) {
      const unsigned j = (unsigned)__builtin_ctz(mask);
      const unsigned patch = (unsigned)((j & 8u ? p23 : p01) >> ((j & 7u) * 8u + 2u)) & 31u;
      *(LDS_AS unsigned short*)ea = (unsigned short)(pos0 | j | (patch << 11));
      ea += dir;
      mask &= mask - 1u;
    }
    if (lane == 0) *(LDS_AS unsigned*)(sc + 2304 + half * 4) = ne | (base << 16);
    BARRIER();
    const unsigned m0 = *(LDS_AS unsigned*)(sc + 2304), m1 = *(LDS_AS unsigned*)(sc + 2308);
    const bool raw = (m0 & 0xFFFFu) + (m1 & 0xFFFFu) > (unsigned)ECAP;
    if (raw) st16(ws + slot + RAW_OFF + (size_t)(G * 2 + half) * 1024 + lane * 16, d);
    if (half == 0 && lane < 28) {
      unsigned v = ((LDS_AS unsigned*)sc)[lane];
      if (lane == 0) v = (m0 >> 16) * 0x04040404u | (raw ? RAW_FLAG : 0u);
      if (lane == 1) v = (m1 >> 16) * 0x04040404u;
      st4(trec + 8 * GRP_P + (G & 7) * GRP_E + lane * 4, v);
    }
    WAITV(0);
    BARRIER();
    if (wid == 0) {
      STAMP(1);
      if (lane < NR) st4(p.ws[lane] + WS_FLAG + ((size_t)p.rank * NWG + b) * 64, call);
      if (lane == 0) __hip_atomic_store(cnt, call, __ATOMIC_RELAXED, __HIP_MEMORY_SCOPE_AGENT);
    }
    if (wid == 5 && __builtin_amdgcn_readfirstlane(lead_old) < call) {
      __builtin_amdgcn_fence(__ATOMIC_ACQUIRE, "");
      WAITV(0);
      if (lane == 0) __hip_atomic_store((unsigned*)(ws + WS_INV + xcc * 128), call, __ATOMIC_RELAXED, __HIP_MEMORY_SCOPE_AGENT);
    }
  }

  const int grp = b & 7, idx = b >> 3;
  const int src = (p.rank + grp) & (NR - 1);
  const int rb = idx >> 3, cb = idx & 7;
  const int col0 = p.rank * N_LOC + cb * 256;

  const int pj = wid & 3;
  unsigned wv[8];
#pragma unroll
  for (int i = 0; i < 8; i++)
    wv[i] = (unsigned)(lane >> 4) * N_ALL + (unsigned)(((lane & 15) ^ ((lane >> 4) | ((i & 1) << 2) | (((i >> 2) & 1) << 3))) * 16) +
            (unsigned)(i * 4 * N_ALL - (i & 3) * 1024);
  const u32x4 wsrd = make_srd(p.w + (size_t)pj * 32 * N_ALL + col0, (unsigned)((size_t)KD * N_ALL - (size_t)pj * 32 * N_ALL - col0));
  const unsigned wm0 = (unsigned)(LDS_W + pj * 8192);
  auto issue_w = [&](int u, int wslot) {
    const unsigned so = (unsigned)u * (128u * N_ALL);
    asm volatile("s_add_u32 m0, %10, %11\n\ts_nop 0\n\t"
                 "buffer_load_dwordx4 %0, %8, %9 offen lds\n\t"
                 "buffer_load_dwordx4 %1, %8, %9 offen offset:1024 lds\n\t"
                 "buffer_load_dwordx4 %2, %8, %9 offen offset:2048 lds\n\t"
                 "buffer_load_dwordx4 %3, %8, %9 offen offset:3072 lds\n\t"
                 "s_add_u32 m0, m0, 4096\n\ts_nop 0\n\t"
                 "buffer_load_dwordx4 %4, %8, %9 offen lds\n\t"
                 "buffer_load_dwordx4 %5, %8, %9 offen offset:1024 lds\n\t"
                 "buffer_load_dwordx4 %6, %8, %9 offen offset:2048 lds\n\t"
                 "buffer_load_dwordx4 %7, %8, %9 offen offset:3072 lds"
                 :: "v"(wv[0]), "v"(wv[1]), "v"(wv[2]), "v"(wv[3]), "v"(wv[4]), "v"(wv[5]), "v"(wv[6]), "v"(wv[7]),
                    "s"(wsrd), "s"(so), "s"(wm0), "n"(wslot * WT_B)
                 : "memory", "scc");
  };
  if (wid >= 4) issue_w(0, 0);

  volatile LDS_AS unsigned* ctl = (volatile LDS_AS unsigned*)(smem + LDS_CTL);
  if (wid == 4) {
    const unsigned* f = (const unsigned*)(ws + WS_FLAG + ((size_t)src * NWG + rb * 64 + lane) * 64);
    const unsigned* inv = (const unsigned*)(ws + WS_INV + xcc * 128);
    unsigned* tmo = (unsigned*)(ws + WS_TMO);
    const unsigned long long t0 = __builtin_amdgcn_s_memrealtime();
    while (true) {
      const unsigned f0 = __hip_atomic_load(f, __ATOMIC_RELAXED, __HIP_MEMORY_SCOPE_SYSTEM);
      const unsigned f2 = __hip_atomic_load(inv, __ATOMIC_RELAXED, __HIP_MEMORY_SCOPE_AGENT);
      if (__all(f0 >= call && f2 >= call)) break;
      if (__hip_atomic_load(tmo, __ATOMIC_RELAXED, __HIP_MEMORY_SCOPE_AGENT) == call ||
          __builtin_amdgcn_s_memrealtime() - t0 > DEADLINE_TICKS) {
        if (lane == 0) __hip_atomic_store(tmo, call, __ATOMIC_RELAXED, __HIP_MEMORY_SCOPE_AGENT);
        break;
      }
      __builtin_amdgcn_s_sleep(1);
    }
    STAMP(2);
    const unsigned t = __hip_atomic_load(tmo, __ATOMIC_RELAXED, __HIP_MEMORY_SCOPE_AGENT);
    if (lane == 0) ctl[0] = (t == call) ? 1u : 0u;
  }
  BARRIER();
  const bool bad = __builtin_amdgcn_readfirstlane(ctl[0]) != 0u;

  const int wn = wid & 3;
  float* const outp = p.out + (size_t)(src * M_SH + rb * 128 + (lane & 15)) * N_LOC + cb * 256 + wn * 64 + (lane >> 4) * 4;

  if (bad) {
    WAITV(0);
    if (wid < 4) {
      const float n = __builtin_nanf("");
      const v4f nv = {n, n, n, n};
      for (int mf = 0; mf < 8; mf++)
        for (int nf = 0; nf < 4; nf++) store_out16(outp + (size_t)(mf * 16) * N_LOC + nf * 16, nv);
    }
    return;
  }

  if (wid >= 4) {
    const uint8_t* const xsrc = p.ws[src] + slot;
    const u32x4 xsrd = make_srd(xsrc + (size_t)rb * NKS * TREC + pj * 2 * GRP_P, (unsigned)(4 * NKS * TREC - rb * NKS * TREC - pj * 2 * GRP_P));
    const unsigned xv = (unsigned)lane * 12u;
    const int el = lane & 31;
    const unsigned ev = (unsigned)(8 * GRP_P + (2 * pj + (lane >> 5)) * GRP_E + (el < 28 ? el : 27) * 4 - pj * 2 * GRP_P);
    const unsigned emask = (el < 2 || el >= 28) ? 0u : 0xFFFFFFFFu;
    const unsigned edef = (el < 2 || el >= 28) ? (unsigned)(el * 8) | ((unsigned)(el * 8 + 4) << 16) : 0u;
    unsigned xw = (unsigned)(LDS_X + pj * 4096 + lane * 16);
    unsigned xg = (unsigned)(LDS_X + (2 * pj + (lane >> 5)) * 2048);
    asm volatile("" : "+v"(xw), "+v"(xg));
#define PPRIO 0
    if (PPRIO) __builtin_amdgcn_s_setprio(PPRIO);
    XRegs X[3];
    auto issue_x = [&](int u, XRegs& r) {
      const unsigned so = (unsigned)u * (unsigned)TREC;
      asm volatile("buffer_load_dwordx3 %0, %5, %7, %8 offen" XPOL "\n\t"
                   "buffer_load_dwordx3 %1, %5, %7, %8 offen offset:768" XPOL "\n\t"
                   "buffer_load_dwordx3 %2, %5, %7, %8 offen offset:1536" XPOL "\n\t"
                   "buffer_load_dwordx3 %3, %5, %7, %8 offen offset:2304" XPOL "\n\t"
                   "buffer_load_dword %4, %6, %7, %8 offen" XPOL
                   : "=&v"(r.p[0]), "=&v"(r.p[1]), "=&v"(r.p[2]), "=&v"(r.p[3]), "=&v"(r.e)
                   : "v"(xv), "v"(ev), "s"(xsrd), "s"(so) : "memory");
    };
    auto unpack = [&](int u, XRegs& r, int xslot) {
      asm volatile("" : "+v"(r.p[0]), "+v"(r.p[1]), "+v"(r.p[2]), "+v"(r.p[3]), "+v"(r.e));
      const unsigned hA0 = (unsigned)__builtin_amdgcn_readlane(r.e, 0), hA1 = (unsigned)__builtin_amdgcn_readlane(r.e, 1);
      const unsigned hB0 = (unsigned)__builtin_amdgcn_readlane(r.e, 32), hB1 = (unsigned)__builtin_amdgcn_readlane(r.e, 33);
      LDS_AS uint8_t* dst = (LDS_AS uint8_t*)(size_t)xw + xslot * XT_B;
      *(LDS_AS u32x4*)(dst)        = unpack12(r.p[0], hA0);
      *(LDS_AS u32x4*)(dst + 1024) = unpack12(r.p[1], hA1);
      *(LDS_AS u32x4*)(dst + 2048) = unpack12(r.p[2], hB0);
      *(LDS_AS u32x4*)(dst + 3072) = unpack12(r.p[3], hB1);
      const unsigned e = (r.e & emask) | edef;
      const unsigned a0 = (e & 0x7FCu) | xg, a1 = ((e >> 16) & 0x7FCu) | xg;
      const unsigned v0 = ((e >> 11) & 31u) << (((e << 3) & 24u) | 2u), v1 = (e >> 27) << (((e >> 13) & 24u) | 2u);
      __hip_atomic_fetch_xor((LDS_AS unsigned*)((LDS_AS uint8_t*)(size_t)a0 + xslot * XT_B), v0, __ATOMIC_RELAXED, __HIP_MEMORY_SCOPE_WORKGROUP);
      __hip_atomic_fetch_xor((LDS_AS unsigned*)((LDS_AS uint8_t*)(size_t)a1 + xslot * XT_B), v1, __ATOMIC_RELAXED, __HIP_MEMORY_SCOPE_WORKGROUP);
      if ((hA0 | hB0) & RAW_FLAG) {
        const uint8_t* g = xsrc + RAW_OFF + ((size_t)(rb * NKS + u) * 16 + pj * 4) * 1024;
#pragma unroll
        for (int k = 0; k < 2; k++) {
          if ((k ? hB0 : hA0) & RAW_FLAG) {
            u32x4 a, c;
            asm volatile("global_load_dwordx4 %0, %2, %3" XPOL "\n\t"
                         "global_load_dwordx4 %1, %2, %3 offset:1024" XPOL "\n\t"
                         "s_waitcnt vmcnt(0)"
                         : "=&v"(a), "=&v"(c) : "v"((unsigned)lane * 16u), "s"(g + k * 2048) : "memory");
            *(LDS_AS u32x4*)(dst + k * 2048) = a;
            *(LDS_AS u32x4*)(dst + k * 2048 + 1024) = c;
          }
        }
      }
    };
    issue_x(0, X[0]); issue_x(1, X[1]);
#define PT(x) do {} while (0)
#define PACC() do {} while (0)
#define PROD(v, k, DO_W, DO_X, NW) do { \
      PT(pt0); \
      if (DO_W) issue_w((v) + 1, ((k) + 1) % 3); \
      if (DO_X) issue_x((v) + 2, X[((k) + 2) % 3]); \
      PT(pt1); \
      NW; \
      PT(pt2); \
      unpack((v), X[(k)], (k)); \
      PT(pt3); \
      BARRIER(); PT(pt4); PACC(); } while (0)
    for (int v0 = 0; v0 < 30; v0 += 3) {
      PROD(v0 + 0, 0, true, true, WAITV(18));
      PROD(v0 + 1, 1, true, true, WAITV(18));
      PROD(v0 + 2, 2, true, true, WAITV(18));
    }
    PROD(30, 0, true, false, WAITV(13));
    PROD(31, 1, false, false, WAITV(0));
    RAW_BARRIER();
    RAW_BARRIER();
    return;
  }

  const float scale = p.sx[0] * p.sw[0];
  const unsigned xaddr = (unsigned)(LDS_X + lane * 16);
  unsigned waddr[4];
  {
    const int g = lane >> 4, q = (lane & 15) >> 1;
    const int f = q | ((g & 1) << 3);
#pragma unroll
    for (int nf = 0; nf < 4; nf++)
      waddr[nf] = (unsigned)(LDS_W + (16 * g + q) * 256 + (((wn * 4 + nf) ^ f) << 4) + (lane & 1) * 8);
  }
#define CPRIO 0
  if (CPRIO) __builtin_amdgcn_s_setprio(CPRIO);
  v8i wf[4], xf[8];
  v4f acc[4][8];
#define READ_W(nf, off) do { LDS_AS uint8_t* wp_ = smem + waddr[nf] + (off); \
    const v2i t0_ = __builtin_amdgcn_ds_read_tr8_b64_v2i32((LDS_AS v2i*)(wp_)); \
    const v2i t1_ = __builtin_amdgcn_ds_read_tr8_b64_v2i32((LDS_AS v2i*)(wp_ + 2048)); \
    const v2i t2_ = __builtin_amdgcn_ds_read_tr8_b64_v2i32((LDS_AS v2i*)(wp_ + 16384)); \
    const v2i t3_ = __builtin_amdgcn_ds_read_tr8_b64_v2i32((LDS_AS v2i*)(wp_ + 18432)); \
    wf[nf] = (v8i){t0_[0], t0_[1], t1_[0], t1_[1], t2_[0], t2_[1], t3_[0], t3_[1]}; } while (0)
#define READ_X(mf, off) do { LDS_AS uint8_t* xp_ = smem + xaddr + (off) + (mf) * 2048; \
    const v4i a_ = *(LDS_AS v4i*)(xp_); const v4i c_ = *(LDS_AS v4i*)(xp_ + 1024); \
    xf[mf] = (v8i){a_[0], a_[1], a_[2], a_[3], c_[0], c_[1], c_[2], c_[3]}; } while (0)
#define MM(nf, mf) acc[nf][mf] = __builtin_amdgcn_mfma_scale_f32_16x16x128_f8f6f4(wf[nf], xf[mf], acc[nf][mf], 1, 1, 0, 0, 0, 0)
#define MROW(mf, xo) do { MM(0, mf); MM(1, mf); MM(2, mf); MM(3, mf); SCHED(); READ_X(mf, xo); SCHED(); } while (0)
#define STEP(xo, wo) do { \
    MROW(0, xo); MROW(1, xo); MROW(2, xo); MROW(3, xo); MROW(4, xo); MROW(5, xo); MROW(6, xo); \
    MM(0, 7); SCHED(); READ_W(0, wo); SCHED(); MM(1, 7); SCHED(); READ_W(1, wo); SCHED(); \
    MM(2, 7); SCHED(); READ_W(2, wo); SCHED(); MM(3, 7); SCHED(); READ_W(3, wo); READ_X(7, xo); SCHED(); \
    BARRIER(); } while (0)

#pragma unroll
  for (int i = 0; i < 4; i++)
#pragma unroll
    for (int j = 0; j < 8; j++) acc[i][j] = (v4f){0.f, 0.f, 0.f, 0.f};
  RAW_BARRIER();
  if (wid == 0) STAMP(4);
#pragma unroll
  for (int i = 0; i < 4; i++) READ_W(i, 0);
#pragma unroll
  for (int i = 0; i < 8; i++) READ_X(i, 0);
  BARRIER();
  int sl = 1;
  for (int u = 0; u < NKS; u++) {
    const int xo = sl * XT_B, wo = sl * WT_B;
    STEP(xo, wo);
    sl = (sl + 1 == 3) ? 0 : sl + 1;
    if (wid == 0 && ((u % 3) == 0 || u == NKS - 1)) STAMP(8 + u);
  }
#pragma unroll
  for (int mf = 0; mf < 8; mf++)
#pragma unroll
    for (int nf = 0; nf < 4; nf++) {
      v4f y = acc[nf][mf] * scale;
      y[0] = fmaxf(y[0], 0.f); y[1] = fmaxf(y[1], 0.f); y[2] = fmaxf(y[2], 0.f); y[3] = fmaxf(y[3], 0.f);
      store_out16(outp + (size_t)(mf * 16) * N_LOC + nf * 16, y);
    }
  if (wid == 0) STAMP(6);
}

}

extern "C" void kernel_launch(int rank, int nranks, ncclComm_t comm,
                              hipStream_t stream,
                              void* const* d_in, const int* in_sizes, int n_in,
                              void* d_out, int out_size,
                              const int* in_dtypes, int out_dtype,
                              void* const* d_out_all,
                              void* const* d_ws_all, size_t ws_bytes) {
  (void)comm; (void)in_sizes; (void)in_dtypes; (void)out_dtype; (void)d_out_all; (void)out_size;
  if (nranks != NR || n_in != 4 || ws_bytes < WS_NEED || (unsigned)rank >= (unsigned)NR) {
    fprintf(stderr, "kernel_launch: unexpected configuration\n");
    return;
  }
  static bool init = false;
  if (!init) {
    HIP_CHECK(hipFuncSetAttribute((const void*)gemm_ag_kernel, hipFuncAttributeMaxDynamicSharedMemorySize, LDS_BYTES));
    init = true;
  }
  Params p;
  p.x = (const uint8_t*)d_in[0]; p.w = (const uint8_t*)d_in[1];
  p.sx = (const float*)d_in[2]; p.sw = (const float*)d_in[3];
  p.out = (float*)d_out;
  for (int i = 0; i < NR; i++) p.ws[i] = (uint8_t*)d_ws_all[i];
  p.rank = rank; p.pad = 0;
  gemm_ag_kernel<<<NWG, NTHR, LDS_BYTES, stream>>>(p);
  HIP_CHECK(hipGetLastError());
}
```

```cpp
#include <hip/hip_runtime.h>
#include <rccl/rccl.h>
#include <cstdio>
#include <cstdint>

#define HIP_CHECK(expr) do { hipError_t _e = (expr); if (_e != hipSuccess) { \
    fprintf(stderr, "HIP error %s at %s:%d\n", hipGetErrorString(_e), __FILE__, __LINE__); \
    return; } } while (0)

namespace {

typedef int      v8i   __attribute__((ext_vector_type(8)));
typedef int      v4i   __attribute__((ext_vector_type(4)));
typedef int      v2i   __attribute__((ext_vector_type(2)));
typedef float    v4f   __attribute__((ext_vector_type(4)));
typedef unsigned u32x4 __attribute__((ext_vector_type(4)));
typedef unsigned u32x3 __attribute__((ext_vector_type(3)));
#define LDS_AS __attribute__((address_space(3)))

constexpr int NR = 8;
constexpr int KD = 4096;
constexpr int N_ALL = 16384;
constexpr int N_LOC = 2048;
constexpr int M_SH = 512;
constexpr int NWG = 256;
constexpr int NTHR = 512;
constexpr int NKS = 32;
constexpr int XT_B = 16384;
constexpr int WT_B = 32768;
constexpr int LDS_X = 0;
constexpr int LDS_CTL = 3 * XT_B;
constexpr int LDS_W = 4 * XT_B;
constexpr int LDS_BYTES = LDS_W + 3 * WT_B;

constexpr int GRP_P = 1536, GRP_E = 112, ECAP = 52;
constexpr int TREC = 8 * (GRP_P + GRP_E);
constexpr unsigned RAW_FLAG = 0x80u;
constexpr int SCR_B = 4608;

constexpr size_t WS_CNT  = 0;
constexpr size_t WS_TMO  = WS_CNT + NWG * 128;
constexpr size_t WS_LEAD = WS_TMO + 128;
constexpr size_t WS_INV  = WS_LEAD + 16 * 128;
constexpr size_t WS_FLAG = WS_INV + 16 * 128 + 3968;
constexpr size_t WS_X    = 262144;
constexpr size_t RAW_OFF = 1703936;
constexpr size_t SLOT_B  = RAW_OFF + (size_t)M_SH * KD;
constexpr size_t WS_NEED = WS_X + 2 * SLOT_B;
static_assert(WS_FLAG % 128 == 0 && WS_FLAG + NR * NWG * 64 <= WS_X, "ws layout");
static_assert(4 * NKS * TREC <= RAW_OFF, "ws layout");

constexpr unsigned long long DEADLINE_TICKS = 30000000ull;

struct Params {
  const uint8_t* x; const uint8_t* w; const float* sx; const float* sw; float* out;
  uint8_t* ws[NR];
  int rank; int pad;
};

#define CONS 0
#define EARLYINV 0
#define PH1 8
#define STAMP(i) do {} while (0)
#define WAITV(n) asm volatile("s_waitcnt vmcnt(%0)" :: "n"(n) : "memory")
#define BARRIER() asm volatile("s_waitcnt lgkmcnt(0)\n\ts_barrier" ::: "memory")
#define RAW_BARRIER() asm volatile("s_barrier" ::: "memory")
#define SCHED() __builtin_amdgcn_sched_barrier(0)

__device__ __forceinline__ void st16(void* p, u32x4 v) { asm volatile("global_store_dwordx4 %0, %1, off sc0 sc1\n\ts_nop 1" :: "v"(p), "v"(v) : "memory"); }
__device__ __forceinline__ void st12(void* p, u32x3 v) { asm volatile("global_store_dwordx3 %0, %1, off sc0 sc1\n\ts_nop 1" :: "v"(p), "v"(v) : "memory"); }
__device__ __forceinline__ void st4(void* p, unsigned v) { asm volatile("global_store_dword %0, %1, off sc0 sc1" :: "v"(p), "v"(v) : "memory"); }
__device__ __forceinline__ void store_out16(void* p, v4f v) { asm volatile("global_store_dwordx4 %0, %1, off sc0 sc1\n\ts_nop 1" :: "v"(p), "v"(v) : "memory"); }

__device__ __forceinline__ unsigned wave_scan(unsigned v) {
  v += __builtin_amdgcn_update_dpp(0u, v, 0x111, 0xf, 0xf, true);
  v += __builtin_amdgcn_update_dpp(0u, v, 0x112, 0xf, 0xf, true);
  v += __builtin_amdgcn_update_dpp(0u, v, 0x114, 0xf, 0xf, true);
  v += __builtin_amdgcn_update_dpp(0u, v, 0x118, 0xf, 0xf, true);
  v += __builtin_amdgcn_update_dpp(0u, v, 0x142, 0xa, 0xf, true);
  v += __builtin_amdgcn_update_dpp(0u, v, 0x143, 0xc, 0xf, true);
  return v;
}

__device__ __forceinline__ u32x4 make_srd(const void* base, unsigned bytes) {
  const unsigned long long a = (unsigned long long)base;
  return (u32x4){(unsigned)__builtin_amdgcn_readfirstlane((unsigned)a), (unsigned)__builtin_amdgcn_readfirstlane((unsigned)(a >> 32)) & 0xFFFFu,
                 (unsigned)__builtin_amdgcn_readfirstlane(bytes), 0x00020000u};
}

__device__ __forceinline__ bool wait_words(const uint8_t* addr, unsigned call, unsigned* tmo, int lane) {
  const unsigned long long t0 = __builtin_amdgcn_s_memrealtime();
  while (true) {
    unsigned v, t;
    asm volatile("global_load_dword %0, %2, off sc0 sc1\n\tglobal_load_dword %1, %3, off sc1\n\ts_waitcnt vmcnt(0)"
                 : "=&v"(v), "=&v"(t) : "v"(addr), "v"(tmo) : "memory");
    if (__all(v >= call)) return true;
    if ((unsigned)__builtin_amdgcn_readfirstlane(t) == call || __builtin_amdgcn_s_memrealtime() - t0 > DEADLINE_TICKS) {
      if (lane == 0) __hip_atomic_store(tmo, call, __ATOMIC_RELAXED, __HIP_MEMORY_SCOPE_AGENT);
      return false;
    }
  }
}

struct XRegs { u32x3 p[4]; unsigned e; };

__global__ __launch_bounds__(NTHR, 2) void gemm_ag_kernel(Params p) {
  extern __shared__ __attribute__((aligned(16))) uint8_t smem_[];
  LDS_AS uint8_t* smem = (LDS_AS uint8_t*)smem_;
  const int tid = threadIdx.x, lane = tid & 63;
  const int wid = __builtin_amdgcn_readfirstlane(tid >> 6);
  const int b = blockIdx.x;
  uint8_t* const ws = p.ws[p.rank];
  if (wid == 0) STAMP(0);

  unsigned* cnt = (unsigned*)(ws + WS_CNT + (size_t)b * 128);
  const int G = b * 4 + (wid >> 1), half = wid & 1;
  const uint8_t* gx = p.x + (size_t)((G >> 8) * 128 + (G & 7) * 16 + (lane & 15)) * KD + ((G >> 3) & 31) * 128 + half * 64 + (lane >> 4) * 16;
  unsigned call_raw; u32x4 d;
  asm volatile("global_load_dwordx4 %0, %2, off\n\tglobal_load_dword %1, %3, off sc1" : "=&v"(d), "=&v"(call_raw) : "v"(gx), "v"(cnt) : "memory");

  const int grp = b & 7, idx = b >> 3;
  const int src = (p.rank + grp) & (NR - 1);
  const int rb = idx >> 3, cb = idx & 7;
  const int col0 = p.rank * N_LOC + cb * 256;

  unsigned long long peer = (unsigned long long)p.ws[0];
#pragma unroll
  for (int i = 1; i < NR; i++) {
    unsigned long long wi = (unsigned long long)p.ws[i];
    asm volatile("" : "+s"(wi));
    peer = (lane == i) ? wi : peer;
  }
  uint8_t* const myflag = (uint8_t*)peer + WS_FLAG + ((size_t)p.rank * NWG + b) * 64;

  LDS_AS uint8_t* sc = smem + LDS_X + (wid >> 1) * SCR_B + 2048;
  if (half == 0 && lane < 28) ((LDS_AS unsigned*)sc)[lane] = (unsigned)(lane * 8) | ((unsigned)(lane * 8 + 4) << 16);
  BARRIER();
  asm volatile("s_waitcnt vmcnt(0)" : "+v"(d), "+v"(call_raw) :: "memory");
  const unsigned call = __builtin_amdgcn_readfirstlane(call_raw) + 1u;
  const size_t slot = WS_X + (size_t)(call & 1u) * SLOT_B;
  const unsigned xcc = (unsigned)__builtin_amdgcn_s_getreg((3 << 11) | 20) & 0xFu;
  unsigned* const tmo = (unsigned*)(ws + WS_TMO);
  unsigned lead_old = call;
  if (wid == 5 && lane == 0)
    asm volatile("global_atomic_umax %0, %1, %2, off sc0" : "+v"(lead_old) : "v"(ws + WS_LEAD + xcc * 128), "v"(call) : "memory");
  {
    const unsigned esum = (unsigned)__builtin_amdgcn_readlane(wave_scan(
        __builtin_amdgcn_udot4((d[0] >> 2) & 0x1F1F1F1Fu, 0x01010101u,
        __builtin_amdgcn_udot4((d[1] >> 2) & 0x1F1F1F1Fu, 0x01010101u,
        __builtin_amdgcn_udot4((d[2] >> 2) & 0x1F1F1F1Fu, 0x01010101u,
        __builtin_amdgcn_udot4((d[3] >> 2) & 0x1F1F1F1Fu, 0x01010101u, 0u, false), false), false), false)), 63);
    const unsigned base = esum < 4280u ? 0u : ((esum - 4280u) >> 10 > 24u ? 24u : (esum - 4280u) >> 10);
    const unsigned b4 = base * 0x04040404u;
    unsigned c[4], P[4];
#pragma unroll
    for (int i = 0; i < 4; i++) {
      const unsigned t = (d[i] | 0x80808080u) - b4;
      c[i] = (d[i] & 0x80808080u) | (t & 0x1F1F1F1Fu);
      P[i] = (c[i] + b4) ^ d[i];
    }
    const u32x3 o = {c[0] | ((c[3] & 0x03030303u) << 5), c[1] | ((c[3] & 0x0C0C0C0Cu) << 3),
                     c[2] | ((c[3] & 0x10101010u) << 1) | ((c[3] & 0x80808080u) >> 1)};
    uint8_t* trec = ws + slot + (size_t)(G >> 3) * TREC;
    st12(trec + (G & 7) * GRP_P + half * 768 + lane * 12, o);
    const unsigned z0 = (P[0] + 0x7C7C7C7Cu) & 0x80808080u, z1 = (P[1] + 0x7C7C7C7Cu) & 0x80808080u;
    const unsigned z2 = (P[2] + 0x7C7C7C7Cu) & 0x80808080u, z3 = (P[3] + 0x7C7C7C7Cu) & 0x80808080u;
    unsigned mask = (__builtin_amdgcn_udot4(z0, 0x08040201u, __builtin_amdgcn_udot4(z1, 0x80402010u, 0u, false), false) >> 7) |
                    (__builtin_amdgcn_udot4(z2, 0x08040201u, __builtin_amdgcn_udot4(z3, 0x80402010u, 0u, false), false) << 1);
    const unsigned ncnt = (unsigned)__builtin_popcount(mask);
    const unsigned incl = wave_scan(ncnt);
    const unsigned ne = (unsigned)__builtin_amdgcn_readlane(incl, 63);
    const int dir = half ? -2 : 2;
    LDS_AS uint8_t* ea = sc + (half ? 110 : 8) + (int)(incl - ncnt) * dir;
    const unsigned long long p01 = ((unsigned long long)P[1] << 32) | P[0], p23 = ((unsigned long long)P[3] << 32) | P[2];
    const unsigned pos0 = ((unsigned)half << 10) | ((unsigned)lane << 4);
    while (mask) {
      const unsigned j = (unsigned)__builtin_ctz(mask);
      const unsigned patch = (unsigned)((j & 8u ? p23 : p01) >> ((j & 7u) * 8u + 2u)) & 31u;
      *(LDS_AS unsigned short*)ea = (unsigned short)(pos0 | j | (patch << 11));
      ea += dir;
      mask &= mask - 1u;
    }
    if (lane == 0) *(LDS_AS unsigned*)(sc + 2304 + half * 4) = ne | (base << 16);
    if (wid == 0) STAMP(101);
    BARRIER();
    const unsigned m0 = *(LDS_AS unsigned*)(sc + 2304), m1 = *(LDS_AS unsigned*)(sc + 2308);
    const bool raw = (m0 & 0xFFFFu) + (m1 & 0xFFFFu) > (unsigned)ECAP;
    if (raw) st16(ws + slot + RAW_OFF + (size_t)(G * 2 + half) * 1024 + lane * 16, d);
    if (half == 0 && lane < 28) {
      unsigned v = ((LDS_AS unsigned*)sc)[lane];
      if (lane == 0) v = (m0 >> 16) * 0x04040404u | (raw ? RAW_FLAG : 0u);
      if (lane == 1) v = (m1 >> 16) * 0x04040404u;
      st4(trec + 8 * GRP_P + (G & 7) * GRP_E + lane * 4, v);
    }
    asm volatile("s_waitcnt vmcnt(0)" : "+v"(lead_old) :: "memory");
    BARRIER();
    if (wid == 0) {
      STAMP(1);
      if (lane < NR) st4(myflag, call);
      if (lane == 0) __hip_atomic_store(cnt, call, __ATOMIC_RELAXED, __HIP_MEMORY_SCOPE_AGENT);
    }
    if (wid == 5 && __builtin_amdgcn_readfirstlane(lead_old) < call) {
      __builtin_amdgcn_fence(__ATOMIC_ACQUIRE, "");
      WAITV(0);
      if (lane == 0) __hip_atomic_store((unsigned*)(ws + WS_INV + xcc * 128), call, __ATOMIC_RELAXED, __HIP_MEMORY_SCOPE_AGENT);
      STAMP(102);
    }
  }

  const int pj = wid & 3;
  unsigned wv[8];
#pragma unroll
  for (int i = 0; i < 8; i++)
    wv[i] = (unsigned)(lane >> 4) * N_ALL + (unsigned)(((lane & 15) ^ ((lane >> 4) | ((i & 1) << 2) | (((i >> 2) & 1) << 3))) * 16) +
            (unsigned)(i * 4 * N_ALL - (i & 3) * 1024);
  const u32x4 wsrd = make_srd(p.w + (size_t)pj * 32 * N_ALL + col0, (unsigned)((size_t)KD * N_ALL - (size_t)pj * 32 * N_ALL - col0));
  const unsigned wm0 = (unsigned)(LDS_W + pj * 8192);
#define W_FIRST(i, so, slotoff) asm volatile("s_add_u32 m0, %3, %4\n\ts_nop 0\n\tbuffer_load_dwordx4 %0, %1, %2 offen lds" \
      :: "v"(wv[i]), "s"(wsrd), "s"(so), "s"(wm0), "n"(slotoff) : "memory", "scc")
#define W_NEXT(i, so) asm volatile("buffer_load_dwordx4 %0, %1, %2 offen offset:%3 lds" :: "v"(wv[i]), "s"(wsrd), "s"(so), "n"(((i) & 3) * 1024) : "memory")
  if (wid >= 4) {
    const unsigned so = 0u;
    W_FIRST(0, so, 0); W_NEXT(1, so); W_NEXT(2, so); W_NEXT(3, so); W_FIRST(4, so, 4096); W_NEXT(5, so); W_NEXT(6, so); W_NEXT(7, so);
  }

  volatile LDS_AS unsigned* ctl = (volatile LDS_AS unsigned*)(smem + LDS_CTL);
  const uint8_t* flagp = ws + WS_FLAG + ((size_t)src * NWG + rb * 64 + lane) * 64;
  if (wid == 4) {
    STAMP(105);
    const bool ok = wait_words(lane < PH1 ? flagp : ws + WS_INV + xcc * 128, call, tmo, lane);
    STAMP(2);
    if (lane == 0) ctl[0] = ok ? 0u : 1u;
  }
  BARRIER();
  const bool bad = __builtin_amdgcn_readfirstlane(ctl[0]) != 0u;

  const int wn = wid & 3;
  float* const outp = p.out + (size_t)(src * M_SH + rb * 128 + (lane & 15)) * N_LOC + cb * 256 + wn * 64 + (lane >> 4) * 4;
  auto store_nan = [&]() {
    const float n = __builtin_nanf("");
    const v4f nv = {n, n, n, n};
    for (int mf = 0; mf < 8; mf++)
      for (int nf = 0; nf < 4; nf++) store_out16(outp + (size_t)(mf * 16) * N_LOC + nf * 16, nv);
  };
  if (bad) {
    WAITV(0);
    if (wid < 4) store_nan();
    return;
  }

  if (wid >= 4) {
    const uint8_t* const xsrc = p.ws[src] + slot;
    u32x4 xsrd = make_srd(xsrc + (size_t)rb * NKS * TREC + pj * 2 * GRP_P, (unsigned)(4 * NKS * TREC - rb * NKS * TREC - pj * 2 * GRP_P));
    const unsigned xv = (unsigned)lane * 12u;
    const int el = lane & 31;
    const unsigned ev = (unsigned)(8 * GRP_P + (2 * pj + (lane >> 5)) * GRP_E + (el < 28 ? el : 27) * 4 - pj * 2 * GRP_P);
    const unsigned emask = (el < 2 || el >= 28) ? 0u : 0xFFFFFFFFu;
    const unsigned edef = (el < 2 || el >= 28) ? (unsigned)(el * 8) | ((unsigned)(el * 8 + 4) << 16) : 0u;
    unsigned xw = (unsigned)(LDS_X + pj * 4096 + lane * 16);
    unsigned xg = (unsigned)(LDS_X + (2 * pj + (lane >> 5)) * 2048);
    asm volatile("" : "+v"(xw), "+v"(xg));
    XRegs X[3];
#define X_LD(r, j, so) asm volatile("buffer_load_dwordx3 %0, %1, %2, %3 offen offset:%4" : "=v"((r).p[j]) : "v"(xv), "s"(xsrd), "s"(so), "n"((j) * 768) : "memory")
#define X_LE(r, so) asm volatile("buffer_load_dword %0, %1, %2, %3 offen" : "=v"((r).e) : "v"(ev), "s"(xsrd), "s"(so) : "memory")
#define X_ALL(r, u) do { const unsigned so_ = (unsigned)(u) * (unsigned)TREC; X_LD(r, 0, so_); X_LD(r, 1, so_); X_LD(r, 2, so_); X_LD(r, 3, so_); X_LE(r, so_); } while (0)
    auto raw_fix = [&](int u, unsigned hA0, unsigned hB0, LDS_AS uint8_t* dst) {
      if (xsrd[2] == 0u) return;
      const uint8_t* g = xsrc + RAW_OFF + ((size_t)(rb * NKS + u) * 16 + pj * 4) * 1024;
#pragma unroll
      for (int k = 0; k < 2; k++) {
        if ((k ? hB0 : hA0) & RAW_FLAG) {
          u32x4 a, c;
          asm volatile("global_load_dwordx4 %0, %2, %3\n\t"
                       "global_load_dwordx4 %1, %2, %3 offset:1024\n\t"
                       "s_waitcnt vmcnt(0)"
                       : "=&v"(a), "=&v"(c) : "v"((unsigned)lane * 16u), "s"(g + k * 2048) : "memory");
          *(LDS_AS u32x4*)(dst + k * 2048) = a;
          *(LDS_AS u32x4*)(dst + k * 2048 + 1024) = c;
        }
      }
    };
    X_ALL(X[0], 0); X_ALL(X[1], 1);
#define PT(x) do {} while (0)
#define PACC() do {} while (0)
#define UNP_LO(o, b) (u32x3){((o)[0] & 0x9F9F9F9Fu) + (b), ((o)[1] & 0x9F9F9F9Fu) + (b), ((o)[2] & 0x9F9F9F9Fu) + (b)}
#define UNP_HI(o, b) ((((o)[0] >> 5) & 0x03030303u) | (((o)[1] >> 3) & 0x0C0C0C0Cu) | (((o)[2] >> 1) & 0x10101010u) | (((o)[2] << 1) & 0x80808080u)) + (b)
#define PACK(j, b, DW, I0, I1) do { \
      const unsigned h_ = UNP_HI(r.p[j], b); SCHED(); if (DW) { I0; } SCHED(); \
      const u32x3 l_ = UNP_LO(r.p[j], b); \
      *(LDS_AS u32x4*)(dst + (j) * 1024) = (u32x4){l_[0], l_[1], l_[2], h_}; SCHED(); if (DW) { I1; } SCHED(); } while (0)
#define PROD(v, k, DO_W, DO_X, NW) do { \
      XRegs& r = X[(k)]; XRegs& rn = X[((k) + 2) % 3]; \
      PT(pt0); NW; PT(pt2); \
      asm volatile("" : "+v"(r.p[0]), "+v"(r.p[1]), "+v"(r.p[2]), "+v"(r.p[3]), "+v"(r.e));       \
      const unsigned soW = (unsigned)((v) + 1) * (128u * N_ALL), soX = (unsigned)((v) + 2) * (unsigned)TREC; \
      const unsigned hA0 = (unsigned)__builtin_amdgcn_readlane(r.e, 0), hA1 = (unsigned)__builtin_amdgcn_readlane(r.e, 1); \
      const unsigned hB0 = (unsigned)__builtin_amdgcn_readlane(r.e, 32), hB1 = (unsigned)__builtin_amdgcn_readlane(r.e, 33); \
      LDS_AS uint8_t* dst = (LDS_AS uint8_t*)(size_t)xw + (k) * XT_B; \
      if (DO_W) W_FIRST(0, soW, (((k) + 1) % 3) * WT_B); SCHED(); \
      PACK(0, hA0, DO_W, W_NEXT(1, soW), W_NEXT(2, soW)); \
      PACK(1, hA1, DO_W, W_NEXT(3, soW), W_FIRST(4, soW, (((k) + 1) % 3) * WT_B + 4096)); \
      PACK(2, hB0, DO_W, W_NEXT(5, soW), W_NEXT(6, soW)); \
      const unsigned e = (r.e & emask) | edef; \
      PACK(3, hB1, DO_W, W_NEXT(7, soW), (void)0); \
      if (DO_X) { X_LD(rn, 0, soX); X_LD(rn, 1, soX); } SCHED(); \
        \
      const unsigned a0 = (e & 0x7FCu) | xg, a1 = ((e >> 16) & 0x7FCu) | xg; \
      const unsigned v0_ = ((e >> 11) & 31u) << (((e << 3) & 24u) | 2u); SCHED(); \
      if (DO_X) { X_LD(rn, 2, soX); } SCHED(); \
      const unsigned v1_ = (e >> 27) << (((e >> 13) & 24u) | 2u); \
      __hip_atomic_fetch_xor((LDS_AS unsigned*)((LDS_AS uint8_t*)(size_t)a0 + (k) * XT_B), v0_, __ATOMIC_RELAXED, __HIP_MEMORY_SCOPE_WORKGROUP); \
      __hip_atomic_fetch_xor((LDS_AS unsigned*)((LDS_AS uint8_t*)(size_t)a1 + (k) * XT_B), v1_, __ATOMIC_RELAXED, __HIP_MEMORY_SCOPE_WORKGROUP); \
      SCHED(); if (DO_X) { X_LD(rn, 3, soX); X_LE(rn, soX); } SCHED(); \
      if ((hA0 | hB0) & RAW_FLAG) raw_fix((v), hA0, hB0, dst); \
      PT(pt3); \
      BARRIER(); PT(pt4); PACC(); } while (0)
    static_assert(PH1 >= 8, "x(2), x(3) are requested before the second wait is published");
    PROD(0, 0, true, true, WAITV(5));
    PROD(1, 1, true, true, WAITV(5));
    if (__builtin_amdgcn_readfirstlane(ctl[1]) != 0u) xsrd[2] = 0u;
    PROD(2, 2, true, true, WAITV(5));
    for (int v0 = 3; v0 < 30; v0 += 3) {
      PROD(v0 + 0, 0, true, true, WAITV(5));
      PROD(v0 + 1, 1, true, true, WAITV(5));
      PROD(v0 + 2, 2, true, true, WAITV(5));
    }
    PROD(30, 0, true, false, WAITV(5));
    PROD(31, 1, false, false, WAITV(0));
    return;
  }

  const float scale = p.sx[0] * p.sw[0];
  const unsigned xaddr = (unsigned)(LDS_X + lane * 16);
  unsigned waddr[4];
  {
    const int g = lane >> 4, q = (lane & 15) >> 1;
    const int f = q | ((g & 1) << 3);
#pragma unroll
    for (int nf = 0; nf < 4; nf++)
      waddr[nf] = (unsigned)(LDS_W + (16 * g + q) * 256 + (((wn * 4 + nf) ^ f) << 4) + (lane & 1) * 8);
  }
  v4f acc[4][8];
#pragma unroll
  for (int i = 0; i < 4; i++)
#pragma unroll
    for (int j = 0; j < 8; j++) acc[i][j] = (v4f){0.f, 0.f, 0.f, 0.f};
#define READ_W(wf, nf, off) do { LDS_AS uint8_t* wp_ = smem + waddr[nf] + (off); \
    const v2i t0_ = __builtin_amdgcn_ds_read_tr8_b64_v2i32((LDS_AS v2i*)(wp_)); \
    const v2i t1_ = __builtin_amdgcn_ds_read_tr8_b64_v2i32((LDS_AS v2i*)(wp_ + 2048)); \
    const v2i t2_ = __builtin_amdgcn_ds_read_tr8_b64_v2i32((LDS_AS v2i*)(wp_ + 16384)); \
    const v2i t3_ = __builtin_amdgcn_ds_read_tr8_b64_v2i32((LDS_AS v2i*)(wp_ + 18432)); \
    wf[nf] = (v8i){t0_[0], t0_[1], t1_[0], t1_[1], t2_[0], t2_[1], t3_[0], t3_[1]}; } while (0)
#define READ_X(r, mf, off) do { LDS_AS uint8_t* xp_ = smem + xaddr + (off) + (mf) * 2048; \
    const v4i a_ = *(LDS_AS v4i*)(xp_); const v4i c_ = *(LDS_AS v4i*)(xp_ + 1024); \
    xf[r] = (v8i){a_[0], a_[1], a_[2], a_[3], c_[0], c_[1], c_[2], c_[3]}; } while (0)
#define CBARRIER(n) asm volatile("s_waitcnt lgkmcnt(%0)\n\ts_barrier" :: "n"(n) : "memory")
  if (wid == 0) {
    const bool ok = wait_words(flagp, call, tmo, lane);
    if (lane == 0) ctl[1] = ok ? 0u : 1u;
  }
  unsigned tmo_late;
  v8i wf[4], xf[8];
#define MM(nf, mf) acc[nf][mf] = __builtin_amdgcn_mfma_scale_f32_16x16x128_f8f6f4(wf[nf], xf[mf], acc[nf][mf], 1, 1, 0, 0, 0, 0)
#define MROW_NR(mf) do { MM(0, mf); MM(1, mf); MM(2, mf); MM(3, mf); } while (0)
#define MROW(mf, xo) do { MROW_NR(mf); SCHED(); READ_X(mf, mf, xo); SCHED(); } while (0)
#define STEP_NB(xo, wo) do { \
    MROW(0, xo); MROW(1, xo); MROW(2, xo); MROW(3, xo); MROW(4, xo); MROW(5, xo); MROW(6, xo); \
    MM(0, 7); SCHED(); READ_W(wf, 0, wo); SCHED(); MM(1, 7); SCHED(); READ_W(wf, 1, wo); SCHED(); \
    MM(2, 7); SCHED(); READ_W(wf, 2, wo); SCHED(); MM(3, 7); SCHED(); READ_W(wf, 3, wo); READ_X(7, 7, xo); SCHED(); } while (0)
  RAW_BARRIER();
  if (wid == 0) STAMP(4);
#pragma unroll
  for (int i = 0; i < 4; i++) READ_W(wf, i, 0);
#pragma unroll
  for (int i = 0; i < 8; i++) READ_X(i, i, 0);
  BARRIER();
  int sl = 1;
  for (int u = 0; u < NKS - 3; u++) {
    STEP_NB(sl * XT_B, sl * WT_B); CBARRIER(0);
    sl = (sl + 1 == 3) ? 0 : sl + 1;
    if (wid == 0 && (u % 6) == 0) STAMP(8 + u);
  }
  asm volatile("global_load_dword %0, %1, off sc1" : "=v"(tmo_late) : "v"(tmo) : "memory");
  {
    STEP_NB(((NKS - 2) % 3) * XT_B, ((NKS - 2) % 3) * WT_B);
    constexpr int xo = ((NKS - 1) % 3) * XT_B, wo = ((NKS - 1) % 3) * WT_B;
    SCHED();
    MROW_NR(0); MROW_NR(1); MROW_NR(2); MROW_NR(3); MROW_NR(4); MROW_NR(5); MROW_NR(6); MROW_NR(7);
    SCHED();
    RAW_BARRIER();
    READ_W(wf, 0, wo); READ_X(0, 0, xo); READ_W(wf, 1, wo); READ_W(wf, 2, wo); READ_W(wf, 3, wo); READ_X(1, 1, xo); SCHED();
    MROW_NR(0); SCHED(); READ_X(2, 2, xo); SCHED(); MROW_NR(1); SCHED(); READ_X(3, 3, xo); SCHED();
    MROW_NR(2); SCHED(); READ_X(4, 4, xo); SCHED(); MROW_NR(3); SCHED(); READ_X(5, 5, xo); SCHED();
    MROW_NR(4); SCHED(); READ_X(6, 6, xo); SCHED(); MROW_NR(5); SCHED(); READ_X(7, 7, xo); SCHED();
    MROW_NR(6); MROW_NR(7); SCHED();
  }
  asm volatile("s_waitcnt vmcnt(0)" : "+v"(tmo_late) :: "memory");
  if (__builtin_amdgcn_readfirstlane(tmo_late) == call) { store_nan(); return; }
#pragma unroll
  for (int mf = 0; mf < 8; mf++)
#pragma unroll
    for (int nf = 0; nf < 4; nf++) {
      v4f y = acc[nf][mf] * scale;
      y[0] = fmaxf(y[0], 0.f); y[1] = fmaxf(y[1], 0.f); y[2] = fmaxf(y[2], 0.f); y[3] = fmaxf(y[3], 0.f);
      store_out16(outp + (size_t)(mf * 16) * N_LOC + nf * 16, y);
    }
}

}

extern "C" void kernel_launch(int rank, int nranks, ncclComm_t comm,
                              hipStream_t stream,
                              void* const* d_in, const int* in_sizes, int n_in,
                              void* d_out, int out_size,
                              const int* in_dtypes, int out_dtype,
                              void* const* d_out_all,
                              void* const* d_ws_all, size_t ws_bytes) {
  (void)comm; (void)in_sizes; (void)in_dtypes; (void)out_dtype; (void)d_out_all; (void)out_size;
  if (nranks != NR || n_in != 4 || ws_bytes < WS_NEED || (unsigned)rank >= (unsigned)NR) {
    fprintf(stderr, "kernel_launch: unexpected configuration\n");
    return;
  }
  static bool init = false;
  if (!init) {
    HIP_CHECK(hipFuncSetAttribute((const void*)gemm_ag_kernel, hipFuncAttributeMaxDynamicSharedMemorySize, LDS_BYTES));
    init = true;
  }
  Params p;
  p.x = (const uint8_t*)d_in[0]; p.w = (const uint8_t*)d_in[1];
  p.sx = (const float*)d_in[2]; p.sw = (const float*)d_in[3];
  p.out = (float*)d_out;
  for (int i = 0; i < NR; i++) p.ws[i] = (uint8_t*)d_ws_all[i];
  p.rank = rank; p.pad = 0;
  gemm_ag_kernel<<<NWG, NTHR, LDS_BYTES, stream>>>(p);
  HIP_CHECK(hipGetLastError());
}
```

```cpp
#include <hip/hip_runtime.h>
#include <rccl/rccl.h>
#include <cstdio>
#include <cstdint>

#define HIP_CHECK(expr) do { hipError_t _e = (expr); if (_e != hipSuccess) { \
    fprintf(stderr, "HIP error %s at %s:%d\n", hipGetErrorString(_e), __FILE__, __LINE__); \
    return; } } while (0)

namespace {

typedef int      v8i   __attribute__((ext_vector_type(8)));
typedef int      v4i   __attribute__((ext_vector_type(4)));
typedef int      v2i   __attribute__((ext_vector_type(2)));
typedef float    v4f   __attribute__((ext_vector_type(4)));
typedef unsigned u32x4 __attribute__((ext_vector_type(4)));
typedef unsigned u32x3 __attribute__((ext_vector_type(3)));
#define LDS_AS __attribute__((address_space(3)))

constexpr int NR = 8;
constexpr int KD = 4096;
constexpr int N_ALL = 16384;
constexpr int N_LOC = 2048;
constexpr int M_SH = 512;
constexpr int NWG = 256;
constexpr int NTHR = 512;
constexpr int NKS = 32;
constexpr int XT_B = 16384;
constexpr int WT_B = 32768;
constexpr int LDS_X = 0;
constexpr int LDS_CTL = 3 * XT_B;
constexpr int LDS_SINK = LDS_CTL + 2048;
constexpr int LDS_W = 4 * XT_B;
constexpr int LDS_BYTES = LDS_W + 3 * WT_B;

constexpr int GRP_P = 1536, GRP_E = 112, ECAP = 52;
constexpr int TREC = 8 * (GRP_P + GRP_E);
constexpr unsigned RAW_FLAG = 0x80u;
constexpr int SCR_B = 4608;

constexpr size_t WS_CNT  = 0;
constexpr size_t WS_TMO  = WS_CNT + NWG * 128;
constexpr size_t WS_LEAD = WS_TMO + 128;
constexpr size_t WS_INV  = WS_LEAD + 16 * 128;
constexpr size_t WS_FLAG = WS_INV + 16 * 128 + 3968;
constexpr size_t WS_X    = 262144;
constexpr size_t RAW_OFF = 1703936;
constexpr size_t SLOT_B  = RAW_OFF + (size_t)M_SH * KD;
constexpr size_t WS_NEED = WS_X + 2 * SLOT_B;
static_assert(WS_FLAG % 128 == 0 && WS_FLAG + NR * NWG * 64 <= WS_X, "ws layout");
static_assert(4 * NKS * TREC <= RAW_OFF, "ws layout");

constexpr unsigned long long DEADLINE_TICKS = 30000000ull;

struct Params {
  const uint8_t* x; const uint8_t* w; const float* sx; const float* sw; float* out;
  uint8_t* ws[NR];
  int rank; int pad;
};

#define CONS 0
#define EARLYINV 0
#define WPF 0
#define KROT 0
#define PH1 8
#define STAMP(i) do {} while (0)
#define WAITV(n) asm volatile("s_waitcnt vmcnt(%0)" :: "n"(n) : "memory")
#define BARRIER() asm volatile("s_waitcnt lgkmcnt(0)\n\ts_barrier" ::: "memory")
#define RAW_BARRIER() asm volatile("s_barrier" ::: "memory")
#define SCHED() __builtin_amdgcn_sched_barrier(0)

__device__ __forceinline__ void st16(void* p, u32x4 v) { asm volatile("global_store_dwordx4 %0, %1, off sc0 sc1\n\ts_nop 1" :: "v"(p), "v"(v) : "memory"); }
__device__ __forceinline__ void st12(void* p, u32x3 v) { asm volatile("global_store_dwordx3 %0, %1, off sc0 sc1\n\ts_nop 1" :: "v"(p), "v"(v) : "memory"); }
__device__ __forceinline__ void st4(void* p, unsigned v) { asm volatile("global_store_dword %0, %1, off sc0 sc1" :: "v"(p), "v"(v) : "memory"); }
__device__ __forceinline__ void store_out16(void* p, v4f v) { asm volatile("global_store_dwordx4 %0, %1, off sc0 sc1\n\ts_nop 1" :: "v"(p), "v"(v) : "memory"); }

__device__ __forceinline__ unsigned wave_scan(unsigned v) {
  v += __builtin_amdgcn_update_dpp(0u, v, 0x111, 0xf, 0xf, true);
  v += __builtin_amdgcn_update_dpp(0u, v, 0x112, 0xf, 0xf, true);
  v += __builtin_amdgcn_update_dpp(0u, v, 0x114, 0xf, 0xf, true);
  v += __builtin_amdgcn_update_dpp(0u, v, 0x118, 0xf, 0xf, true);
  v += __builtin_amdgcn_update_dpp(0u, v, 0x142, 0xa, 0xf, true);
  v += __builtin_amdgcn_update_dpp(0u, v, 0x143, 0xc, 0xf, true);
  return v;
}

__device__ __forceinline__ u32x4 make_srd(const void* base, unsigned bytes) {
  const unsigned long long a = (unsigned long long)base;
  return (u32x4){(unsigned)__builtin_amdgcn_readfirstlane((unsigned)a), (unsigned)__builtin_amdgcn_readfirstlane((unsigned)(a >> 32)) & 0xFFFFu,
                 (unsigned)__builtin_amdgcn_readfirstlane(bytes), 0x00020000u};
}

__device__ __forceinline__ bool wait_words(const uint8_t* addr, unsigned call, unsigned* tmo, int lane) {
  const unsigned long long t0 = __builtin_amdgcn_s_memrealtime();
  while (true) {
    unsigned v, t;
    asm volatile("global_load_dword %0, %2, off sc0 sc1\n\tglobal_load_dword %1, %3, off sc1\n\ts_waitcnt vmcnt(0)"
                 : "=&v"(v), "=&v"(t) : "v"(addr), "v"(tmo) : "memory");
    if (__all(v >= call)) return true;
    if ((unsigned)__builtin_amdgcn_readfirstlane(t) == call || __builtin_amdgcn_s_memrealtime() - t0 > DEADLINE_TICKS) {
      if (lane == 0) __hip_atomic_store(tmo, call, __ATOMIC_RELAXED, __HIP_MEMORY_SCOPE_AGENT);
      return false;
    }
  }
}

struct XRegs { u32x3 p[4]; unsigned e; };

__global__ __launch_bounds__(NTHR, 2) void gemm_ag_kernel(Params p) {
  extern __shared__ __attribute__((aligned(16))) uint8_t smem_[];
  LDS_AS uint8_t* smem = (LDS_AS uint8_t*)smem_;
  const int tid = threadIdx.x, lane = tid & 63;
  const int wid = __builtin_amdgcn_readfirstlane(tid >> 6);
  const int b = blockIdx.x;
  uint8_t* const ws = p.ws[p.rank];
  if (wid == 0) STAMP(0);

  unsigned* cnt = (unsigned*)(ws + WS_CNT + (size_t)b * 128);
  const int G = b * 4 + (wid >> 1), half = wid & 1;
  const uint8_t* gx = p.x + (size_t)((G >> 8) * 128 + (G & 7) * 16 + (lane & 15)) * KD + ((G >> 3) & 31) * 128 + half * 64 + (lane >> 4) * 16;
  unsigned call_raw; u32x4 d;
  asm volatile("global_load_dwordx4 %0, %2, off\n\tglobal_load_dword %1, %3, off sc1" : "=&v"(d), "=&v"(call_raw) : "v"(gx), "v"(cnt) : "memory");

#define W_BULK() do { \
    const u32x4 psrd_ = make_srd(p.w + (size_t)(b * 16 + wid * 2) * N_ALL + p.rank * N_LOC, (unsigned)(N_ALL + N_LOC)); \
    const unsigned pv_ = (unsigned)lane * 16u; \
    asm volatile("s_mov_b32 m0, %2\n\ts_nop 0\n\t" \
                 "buffer_load_dwordx4 %0, %1, 0 offen lds\n\tbuffer_load_dwordx4 %0, %1, 0 offen offset:1024 lds\n\t" \
                 "buffer_load_dwordx4 %0, %1, %3 offen lds\n\tbuffer_load_dwordx4 %0, %1, %3 offen offset:1024 lds" \
                 :: "v"(pv_), "s"(psrd_), "s"((unsigned)LDS_SINK), "s"((unsigned)N_ALL) : "memory"); } while (0)
  const int grp = b & 7, idx = b >> 3;
  const int src = (p.rank + grp) & (NR - 1);
  const int rb = idx >> 3, cb = idx & 7;
  const int col0 = p.rank * N_LOC + cb * 256;
  const unsigned krot = (unsigned)(grp * KROT);
#define KST(u) (((unsigned)(u) + krot) & 31u)

  unsigned long long peer = (unsigned long long)p.ws[0];
#pragma unroll
  for (int i = 1; i < NR; i++) {
    unsigned long long wi = (unsigned long long)p.ws[i];
    asm volatile("" : "+s"(wi));
    peer = (lane == i) ? wi : peer;
  }
  uint8_t* const myflag = (uint8_t*)peer + WS_FLAG + ((size_t)p.rank * NWG + b) * 64;

  LDS_AS uint8_t* sc = smem + LDS_X + (wid >> 1) * SCR_B + 2048;
  if (half == 0 && lane < 28) ((LDS_AS unsigned*)sc)[lane] = (unsigned)(lane * 8) | ((unsigned)(lane * 8 + 4) << 16);
  BARRIER();
  asm volatile("s_waitcnt vmcnt(%2)" : "+v"(d), "+v"(call_raw) : "n"(WPF == 1 ? 4 : 0) : "memory");
  const unsigned call = __builtin_amdgcn_readfirstlane(call_raw) + 1u;
  const size_t slot = WS_X + (size_t)(call & 1u) * SLOT_B;
  const unsigned xcc = (unsigned)__builtin_amdgcn_s_getreg((3 << 11) | 20) & 0xFu;
  unsigned* const tmo = (unsigned*)(ws + WS_TMO);
  unsigned lead_old = call;
  if (wid == 5 && lane == 0)
    asm volatile("global_atomic_umax %0, %1, %2, off sc0" : "+v"(lead_old) : "v"(ws + WS_LEAD + xcc * 128), "v"(call) : "memory");
  {
    const unsigned esum = (unsigned)__builtin_amdgcn_readlane(wave_scan(
        __builtin_amdgcn_udot4((d[0] >> 2) & 0x1F1F1F1Fu, 0x01010101u,
        __builtin_amdgcn_udot4((d[1] >> 2) & 0x1F1F1F1Fu, 0x01010101u,
        __builtin_amdgcn_udot4((d[2] >> 2) & 0x1F1F1F1Fu, 0x01010101u,
        __builtin_amdgcn_udot4((d[3] >> 2) & 0x1F1F1F1Fu, 0x01010101u, 0u, false), false), false), false)), 63);
    const unsigned base = esum < 4280u ? 0u : ((esum - 4280u) >> 10 > 24u ? 24u : (esum - 4280u) >> 10);
    const unsigned b4 = base * 0x04040404u;
    unsigned c[4], P[4];
#pragma unroll
    for (int i = 0; i < 4; i++) {
      const unsigned t = (d[i] | 0x80808080u) - b4;
      c[i] = (d[i] & 0x80808080u) | (t & 0x1F1F1F1Fu);
      P[i] = (c[i] + b4) ^ d[i];
    }
    const u32x3 o = {c[0] | ((c[3] & 0x03030303u) << 5), c[1] | ((c[3] & 0x0C0C0C0Cu) << 3),
                     c[2] | ((c[3] & 0x10101010u) << 1) | ((c[3] & 0x80808080u) >> 1)};
    uint8_t* trec = ws + slot + (size_t)(G >> 3) * TREC;
    st12(trec + (G & 7) * GRP_P + half * 768 + lane * 12, o);
    const unsigned z0 = (P[0] + 0x7C7C7C7Cu) & 0x80808080u, z1 = (P[1] + 0x7C7C7C7Cu) & 0x80808080u;
    const unsigned z2 = (P[2] + 0x7C7C7C7Cu) & 0x80808080u, z3 = (P[3] + 0x7C7C7C7Cu) & 0x80808080u;
    unsigned mask = (__builtin_amdgcn_udot4(z0, 0x08040201u, __builtin_amdgcn_udot4(z1, 0x80402010u, 0u, false), false) >> 7) |
                    (__builtin_amdgcn_udot4(z2, 0x08040201u, __builtin_amdgcn_udot4(z3, 0x80402010u, 0u, false), false) << 1);
    const unsigned ncnt = (unsigned)__builtin_popcount(mask);
    const unsigned incl = wave_scan(ncnt);
    const unsigned ne = (unsigned)__builtin_amdgcn_readlane(incl, 63);
    const int dir = half ? -2 : 2;
    LDS_AS uint8_t* ea = sc + (half ? 110 : 8) + (int)(incl - ncnt) * dir;
    const unsigned long long p01 = ((unsigned long long)P[1] << 32) | P[0], p23 = ((unsigned long long)P[3] << 32) | P[2];
    const unsigned pos0 = ((unsigned)half << 10) | ((unsigned)lane << 4);
    while (mask) {
      const unsigned j = (unsigned)__builtin_ctz(mask);
      const unsigned patch = (unsigned)((j & 8u ? p23 : p01) >> ((j & 7u) * 8u + 2u)) & 31u;
      *(LDS_AS unsigned short*)ea = (unsigned short)(pos0 | j | (patch << 11));
      ea += dir;
      mask &= mask - 1u;
    }
    if (lane == 0) *(LDS_AS unsigned*)(sc + 2304 + half * 4) = ne | (base << 16);
    if (wid == 0) STAMP(101);
    BARRIER();
    const unsigned m0 = *(LDS_AS unsigned*)(sc + 2304), m1 = *(LDS_AS unsigned*)(sc + 2308);
    const bool raw = (m0 & 0xFFFFu) + (m1 & 0xFFFFu) > (unsigned)ECAP;
    if (raw) st16(ws + slot + RAW_OFF + (size_t)(G * 2 + half) * 1024 + lane * 16, d);
    if (half == 0 && lane < 28) {
      unsigned v = ((LDS_AS unsigned*)sc)[lane];
      if (lane == 0) v = (m0 >> 16) * 0x04040404u | (raw ? RAW_FLAG : 0u);
      if (lane == 1) v = (m1 >> 16) * 0x04040404u;
      st4(trec + 8 * GRP_P + (G & 7) * GRP_E + lane * 4, v);
    }
    asm volatile("s_waitcnt vmcnt(0)" : "+v"(lead_old) :: "memory");
    BARRIER();
    if (wid == 0) {
      STAMP(1);
      if (lane < NR) st4(myflag, call);
      if (lane == 0) __hip_atomic_store(cnt, call, __ATOMIC_RELAXED, __HIP_MEMORY_SCOPE_AGENT);
    }
    if (wid == 5 && __builtin_amdgcn_readfirstlane(lead_old) < call) {
      __builtin_amdgcn_fence(__ATOMIC_ACQUIRE, "");
      WAITV(0);
      if (lane == 0) __hip_atomic_store((unsigned*)(ws + WS_INV + xcc * 128), call, __ATOMIC_RELAXED, __HIP_MEMORY_SCOPE_AGENT);
      STAMP(102);
    }
  }

  const int pj = wid & 3;
  unsigned wv[8];
#pragma unroll
  for (int i = 0; i < 8; i++)
    wv[i] = (unsigned)(lane >> 4) * N_ALL + (unsigned)(((lane & 15) ^ ((lane >> 4) | ((i & 1) << 2) | (((i >> 2) & 1) << 3))) * 16) +
            (unsigned)(i * 4 * N_ALL - (i & 3) * 1024);
  const u32x4 wsrd = make_srd(p.w + (size_t)pj * 32 * N_ALL + col0, (unsigned)((size_t)KD * N_ALL - (size_t)pj * 32 * N_ALL - col0));
  const unsigned wm0 = (unsigned)(LDS_W + pj * 8192);
#define W_FIRST(i, so, slotoff) asm volatile("s_add_u32 m0, %3, %4\n\ts_nop 0\n\tbuffer_load_dwordx4 %0, %1, %2 offen lds" \
      :: "v"(wv[i]), "s"(wsrd), "s"(so), "s"(wm0), "n"(slotoff) : "memory", "scc")
#define W_NEXT(i, so) asm volatile("buffer_load_dwordx4 %0, %1, %2 offen offset:%3 lds" :: "v"(wv[i]), "s"(wsrd), "s"(so), "n"(((i) & 3) * 1024) : "memory")
  if (wid >= 4) {
    const unsigned so = KST(0) * (128u * N_ALL);
    W_FIRST(0, so, 0); W_NEXT(1, so); W_NEXT(2, so); W_NEXT(3, so); W_FIRST(4, so, 4096); W_NEXT(5, so); W_NEXT(6, so); W_NEXT(7, so);
  }

  volatile LDS_AS unsigned* ctl = (volatile LDS_AS unsigned*)(smem + LDS_CTL);
  const uint8_t* flagp = ws + WS_FLAG + ((size_t)src * NWG + rb * 64 + lane) * 64;
  if (wid == 4) {
    STAMP(105);
    const bool ok = wait_words((((unsigned)lane - 2u * krot) & 63u) < PH1 ? flagp : ws + WS_INV + xcc * 128, call, tmo, lane);
    STAMP(2);
    if (lane == 0) ctl[0] = ok ? 0u : 1u;
  }
  BARRIER();
  const bool bad = __builtin_amdgcn_readfirstlane(ctl[0]) != 0u;

  const int wn = wid & 3;
  float* const outp = p.out + (size_t)(src * M_SH + rb * 128 + (lane & 15)) * N_LOC + cb * 256 + wn * 64 + (lane >> 4) * 4;
  auto store_nan = [&]() {
    const float n = __builtin_nanf("");
    const v4f nv = {n, n, n, n};
    for (int mf = 0; mf < 8; mf++)
      for (int nf = 0; nf < 4; nf++) store_out16(outp + (size_t)(mf * 16) * N_LOC + nf * 16, nv);
  };
  if (bad) {
    WAITV(0);
    if (wid < 4) store_nan();
    return;
  }

  if (wid >= 4) {
    const uint8_t* const xsrc = p.ws[src] + slot;
    u32x4 xsrd = make_srd(xsrc + (size_t)rb * NKS * TREC + pj * 2 * GRP_P, (unsigned)(4 * NKS * TREC - rb * NKS * TREC - pj * 2 * GRP_P));
    const unsigned xv = (unsigned)lane * 12u;
    const int el = lane & 31;
    const unsigned ev = (unsigned)(8 * GRP_P + (2 * pj + (lane >> 5)) * GRP_E + (el < 28 ? el : 27) * 4 - pj * 2 * GRP_P);
    const unsigned emask = (el < 2 || el >= 28) ? 0u : 0xFFFFFFFFu;
    const unsigned edef = (el < 2 || el >= 28) ? (unsigned)(el * 8) | ((unsigned)(el * 8 + 4) << 16) : 0u;
    unsigned xw = (unsigned)(LDS_X + pj * 4096 + lane * 16);
    unsigned xg = (unsigned)(LDS_X + (2 * pj + (lane >> 5)) * 2048);
    asm volatile("" : "+v"(xw), "+v"(xg));
    XRegs X[3];
#define X_LD(r, j, so) asm volatile("buffer_load_dwordx3 %0, %1, %2, %3 offen offset:%4" : "=v"((r).p[j]) : "v"(xv), "s"(xsrd), "s"(so), "n"((j) * 768) : "memory")
#define X_LE(r, so) asm volatile("buffer_load_dword %0, %1, %2, %3 offen" : "=v"((r).e) : "v"(ev), "s"(xsrd), "s"(so) : "memory")
#define X_ALL(r, u) do { const unsigned so_ = KST(u) * (unsigned)TREC; X_LD(r, 0, so_); X_LD(r, 1, so_); X_LD(r, 2, so_); X_LD(r, 3, so_); X_LE(r, so_); } while (0)
    auto raw_fix = [&](int u, unsigned hA0, unsigned hB0, LDS_AS uint8_t* dst) {
      if (xsrd[2] == 0u) return;
      const uint8_t* g = xsrc + RAW_OFF + ((size_t)(rb * NKS + KST(u)) * 16 + pj * 4) * 1024;
#pragma unroll
      for (int k = 0; k < 2; k++) {
        if ((k ? hB0 : hA0) & RAW_FLAG) {
          u32x4 a, c;
          asm volatile("global_load_dwordx4 %0, %2, %3\n\t"
                       "global_load_dwordx4 %1, %2, %3 offset:1024\n\t"
                       "s_waitcnt vmcnt(0)"
                       : "=&v"(a), "=&v"(c) : "v"((unsigned)lane * 16u), "s"(g + k * 2048) : "memory");
          *(LDS_AS u32x4*)(dst + k * 2048) = a;
          *(LDS_AS u32x4*)(dst + k * 2048 + 1024) = c;
        }
      }
    };
    X_ALL(X[0], 0); X_ALL(X[1], 1);
#define PT(x) do {} while (0)
#define PACC() do {} while (0)
#define UNP_LO(o, b) (u32x3){((o)[0] & 0x9F9F9F9Fu) + (b), ((o)[1] & 0x9F9F9F9Fu) + (b), ((o)[2] & 0x9F9F9F9Fu) + (b)}
#define UNP_HI(o, b) ((((o)[0] >> 5) & 0x03030303u) | (((o)[1] >> 3) & 0x0C0C0C0Cu) | (((o)[2] >> 1) & 0x10101010u) | (((o)[2] << 1) & 0x80808080u)) + (b)
#define PACK(j, b, DW, I0, I1) do { \
      const unsigned h_ = UNP_HI(r.p[j], b); SCHED(); if (DW) { I0; } SCHED(); \
      const u32x3 l_ = UNP_LO(r.p[j], b); \
      *(LDS_AS u32x4*)(dst + (j) * 1024) = (u32x4){l_[0], l_[1], l_[2], h_}; SCHED(); if (DW) { I1; } SCHED(); } while (0)
#define WPOS 3
#define W_I(i, so, off) do { if ((i) == 0) W_FIRST(0, so, off); else if ((i) == 4) W_FIRST(4, so, (off) + 4096); else W_NEXT(i, so); } while (0)
#define WPART0(DW, so, off) (void)0
#define WP(s, so, off) (void)0
#define PROD(v, k, DO_W, DO_X, NW) do { \
      XRegs& r = X[(k)]; XRegs& rn = X[((k) + 2) % 3]; \
      const unsigned soW = KST((v) + 1) * (128u * N_ALL), soX = KST((v) + 2) * (unsigned)TREC; \
      if (WPOS == 3) { \
        if (DO_W) { W_I(0, soW, (((k) + 1) % 3) * WT_B); W_I(1, soW, 0); W_I(2, soW, 0); W_I(3, soW, 0); W_I(4, soW, (((k) + 1) % 3) * WT_B); W_I(5, soW, 0); W_I(6, soW, 0); W_I(7, soW, 0); } \
        if (DO_X) { X_LD(rn, 0, soX); X_LD(rn, 1, soX); X_LD(rn, 2, soX); X_LD(rn, 3, soX); X_LE(rn, soX); } \
      } \
      PT(pt0); NW; PT(pt2); \
      asm volatile("" : "+v"(r.p[0]), "+v"(r.p[1]), "+v"(r.p[2]), "+v"(r.p[3]), "+v"(r.e));       \
      const unsigned hA0 = (unsigned)__builtin_amdgcn_readlane(r.e, 0), hA1 = (unsigned)__builtin_amdgcn_readlane(r.e, 1); \
      const unsigned hB0 = (unsigned)__builtin_amdgcn_readlane(r.e, 32), hB1 = (unsigned)__builtin_amdgcn_readlane(r.e, 33); \
      LDS_AS uint8_t* dst = (LDS_AS uint8_t*)(size_t)xw + (k) * XT_B; \
      WPART0(DO_W, soW, (((k) + 1) % 3) * WT_B); \
      PACK(0, hA0, DO_W, WP(1, soW, (((k) + 1) % 3) * WT_B), WP(2, soW, (((k) + 1) % 3) * WT_B)); \
      PACK(1, hA1, DO_W, WP(3, soW, (((k) + 1) % 3) * WT_B), WP(4, soW, (((k) + 1) % 3) * WT_B)); \
      PACK(2, hB0, DO_W, WP(5, soW, (((k) + 1) % 3) * WT_B), WP(6, soW, (((k) + 1) % 3) * WT_B)); \
      const unsigned e = (r.e & emask) | edef; \
      PACK(3, hB1, DO_W, WP(7, soW, (((k) + 1) % 3) * WT_B), (void)0); \
      if (DO_X && WPOS != 3) { X_LD(rn, 0, soX); X_LD(rn, 1, soX); } SCHED(); \
        \
      const unsigned a0 = (e & 0x7FCu) | xg, a1 = ((e >> 16) & 0x7FCu) | xg; \
      const unsigned v0_ = ((e >> 11) & 31u) << (((e << 3) & 24u) | 2u); SCHED(); \
      if (DO_X && WPOS != 3) { X_LD(rn, 2, soX); } SCHED(); \
      const unsigned v1_ = (e >> 27) << (((e >> 13) & 24u) | 2u); \
      __hip_atomic_fetch_xor((LDS_AS unsigned*)((LDS_AS uint8_t*)(size_t)a0 + (k) * XT_B), v0_, __ATOMIC_RELAXED, __HIP_MEMORY_SCOPE_WORKGROUP); \
      __hip_atomic_fetch_xor((LDS_AS unsigned*)((LDS_AS uint8_t*)(size_t)a1 + (k) * XT_B), v1_, __ATOMIC_RELAXED, __HIP_MEMORY_SCOPE_WORKGROUP); \
      SCHED(); if (DO_X && WPOS != 3) { X_LD(rn, 3, soX); X_LE(rn, soX); } SCHED(); \
      if ((hA0 | hB0) & RAW_FLAG) raw_fix((v), hA0, hB0, dst); \
      PT(pt3); \
      BARRIER(); PT(pt4); PACC(); } while (0)
    constexpr int NWT = WPOS == 3 ? 18 : 5;
    static_assert(PH1 >= 8, "x(2), x(3) are requested before the second wait is published");
    PROD(0, 0, true, true, WAITV(NWT));
    PROD(1, 1, true, true, WAITV(NWT));
    if (__builtin_amdgcn_readfirstlane(ctl[1]) != 0u) xsrd[2] = 0u;
    PROD(2, 2, true, true, WAITV(NWT));
    for (int v0 = 3; v0 < 30; v0 += 3) {
      PROD(v0 + 0, 0, true, true, WAITV(NWT));
      PROD(v0 + 1, 1, true, true, WAITV(NWT));
      PROD(v0 + 2, 2, true, true, WAITV(NWT));
    }
    PROD(30, 0, true, false, WAITV(WPOS == 3 ? 13 : 5));
    PROD(31, 1, false, false, WAITV(0));
    return;
  }

  const float scale = p.sx[0] * p.sw[0];
  const unsigned xaddr = (unsigned)(LDS_X + lane * 16);
  unsigned waddr[4];
  {
    const int g = lane >> 4, q = (lane & 15) >> 1;
    const int f = q | ((g & 1) << 3);
#pragma unroll
    for (int nf = 0; nf < 4; nf++)
      waddr[nf] = (unsigned)(LDS_W + (16 * g + q) * 256 + (((wn * 4 + nf) ^ f) << 4) + (lane & 1) * 8);
  }
  v4f acc[4][8];
#pragma unroll
  for (int i = 0; i < 4; i++)
#pragma unroll
    for (int j = 0; j < 8; j++) acc[i][j] = (v4f){0.f, 0.f, 0.f, 0.f};
#define READ_W(wf, nf, off) do { LDS_AS uint8_t* wp_ = smem + waddr[nf] + (off); \
    const v2i t0_ = __builtin_amdgcn_ds_read_tr8_b64_v2i32((LDS_AS v2i*)(wp_)); \
    const v2i t1_ = __builtin_amdgcn_ds_read_tr8_b64_v2i32((LDS_AS v2i*)(wp_ + 2048)); \
    const v2i t2_ = __builtin_amdgcn_ds_read_tr8_b64_v2i32((LDS_AS v2i*)(wp_ + 16384)); \
    const v2i t3_ = __builtin_amdgcn_ds_read_tr8_b64_v2i32((LDS_AS v2i*)(wp_ + 18432)); \
    wf[nf] = (v8i){t0_[0], t0_[1], t1_[0], t1_[1], t2_[0], t2_[1], t3_[0], t3_[1]}; } while (0)
#define READ_X(r, mf, off) do { LDS_AS uint8_t* xp_ = smem + xaddr + (off) + (mf) * 2048; \
    const v4i a_ = *(LDS_AS v4i*)(xp_); const v4i c_ = *(LDS_AS v4i*)(xp_ + 1024); \
    xf[r] = (v8i){a_[0], a_[1], a_[2], a_[3], c_[0], c_[1], c_[2], c_[3]}; } while (0)
#define CBARRIER(n) asm volatile("s_waitcnt lgkmcnt(%0)\n\ts_barrier" :: "n"(n) : "memory")
  if (wid == 0) {
    const bool ok = wait_words(flagp, call, tmo, lane);
    if (lane == 0) ctl[1] = ok ? 0u : 1u;
  }
  unsigned tmo_late;
  v8i wf[4], xf[8];
#define MM(nf, mf) acc[nf][mf] = __builtin_amdgcn_mfma_scale_f32_16x16x128_f8f6f4(wf[nf], xf[mf], acc[nf][mf], 1, 1, 0, 0, 0, 0)
#define MROW_NR(mf) do { MM(0, mf); MM(1, mf); MM(2, mf); MM(3, mf); } while (0)
#define MROW(mf, xo) do { MROW_NR(mf); SCHED(); READ_X(mf, mf, xo); SCHED(); } while (0)
#define STEP_NB(xo, wo) do { \
    MROW(0, xo); MROW(1, xo); MROW(2, xo); MROW(3, xo); MROW(4, xo); MROW(5, xo); MROW(6, xo); \
    MM(0, 7); SCHED(); READ_W(wf, 0, wo); SCHED(); MM(1, 7); SCHED(); READ_W(wf, 1, wo); SCHED(); \
    MM(2, 7); SCHED(); READ_W(wf, 2, wo); SCHED(); MM(3, 7); SCHED(); READ_W(wf, 3, wo); READ_X(7, 7, xo); SCHED(); } while (0)
  RAW_BARRIER();
  if (wid == 0) STAMP(4);
#pragma unroll
  for (int i = 0; i < 4; i++) READ_W(wf, i, 0);
#pragma unroll
  for (int i = 0; i < 8; i++) READ_X(i, i, 0);
  BARRIER();
  int sl = 1;
  for (int u = 0; u < NKS - 3; u++) {
    STEP_NB(sl * XT_B, sl * WT_B); CBARRIER(0);
    sl = (sl + 1 == 3) ? 0 : sl + 1;
    if (wid == 0 && (u % 6) == 0) STAMP(8 + u);
  }
  asm volatile("global_load_dword %0, %1, off sc1" : "=v"(tmo_late) : "v"(tmo) : "memory");
  {
    STEP_NB(((NKS - 2) % 3) * XT_B, ((NKS - 2) % 3) * WT_B);
    constexpr int xo = ((NKS - 1) % 3) * XT_B, wo = ((NKS - 1) % 3) * WT_B;
    SCHED();
    MROW_NR(0); MROW_NR(1); MROW_NR(2); MROW_NR(3); MROW_NR(4); MROW_NR(5); MROW_NR(6); MROW_NR(7);
    SCHED();
    RAW_BARRIER();
    READ_W(wf, 0, wo); READ_X(0, 0, xo); READ_W(wf, 1, wo); READ_W(wf, 2, wo); READ_W(wf, 3, wo); READ_X(1, 1, xo); SCHED();
    MROW_NR(0); SCHED(); READ_X(2, 2, xo); SCHED(); MROW_NR(1); SCHED(); READ_X(3, 3, xo); SCHED();
    MROW_NR(2); SCHED(); READ_X(4, 4, xo); SCHED(); MROW_NR(3); SCHED(); READ_X(5, 5, xo); SCHED();
    MROW_NR(4); SCHED(); READ_X(6, 6, xo); SCHED(); MROW_NR(5); SCHED(); READ_X(7, 7, xo); SCHED();
    MROW_NR(6); MROW_NR(7); SCHED();
  }
  asm volatile("s_waitcnt vmcnt(0)" : "+v"(tmo_late) :: "memory");
  if (__builtin_amdgcn_readfirstlane(tmo_late) == call) { store_nan(); return; }
#pragma unroll
  for (int mf = 0; mf < 8; mf++)
#pragma unroll
    for (int nf = 0; nf < 4; nf++) {
      v4f y = acc[nf][mf] * scale;
      y[0] = fmaxf(y[0], 0.f); y[1] = fmaxf(y[1], 0.f); y[2] = fmaxf(y[2], 0.f); y[3] = fmaxf(y[3], 0.f);
      store_out16(outp + (size_t)(mf * 16) * N_LOC + nf * 16, y);
    }
}

}

extern "C" void kernel_launch(int rank, int nranks, ncclComm_t comm,
                              hipStream_t stream,
                              void* const* d_in, const int* in_sizes, int n_in,
                              void* d_out, int out_size,
                              const int* in_dtypes, int out_dtype,
                              void* const* d_out_all,
                              void* const* d_ws_all, size_t ws_bytes) {
  (void)comm; (void)in_sizes; (void)in_dtypes; (void)out_dtype; (void)d_out_all; (void)out_size;
  if (nranks != NR || n_in != 4 || ws_bytes < WS_NEED || (unsigned)rank >= (unsigned)NR) {
    fprintf(stderr, "kernel_launch: unexpected configuration\n");
    return;
  }
  static bool init = false;
  if (!init) {
    HIP_CHECK(hipFuncSetAttribute((const void*)gemm_ag_kernel, hipFuncAttributeMaxDynamicSharedMemorySize, LDS_BYTES));
    init = true;
  }
  Params p;
  p.x = (const uint8_t*)d_in[0]; p.w = (const uint8_t*)d_in[1];
  p.sx = (const float*)d_in[2]; p.sw = (const float*)d_in[3];
  p.out = (float*)d_out;
  for (int i = 0; i < NR; i++) p.ws[i] = (uint8_t*)d_ws_all[i];
  p.rank = rank; p.pad = 0;
  gemm_ag_kernel<<<NWG, NTHR, LDS_BYTES, stream>>>(p);
  HIP_CHECK(hipGetLastError());
}
```

```cpp
#include <hip/hip_runtime.h>
#include <rccl/rccl.h>
#include <cstdio>
#include <cstdint>

#define HIP_CHECK(expr) do { hipError_t _e = (expr); if (_e != hipSuccess) { \
    fprintf(stderr, "HIP error %s at %s:%d\n", hipGetErrorString(_e), __FILE__, __LINE__); \
    return; } } while (0)

namespace {

typedef int      v8i   __attribute__((ext_vector_type(8)));
typedef int      v4i   __attribute__((ext_vector_type(4)));
typedef int      v2i   __attribute__((ext_vector_type(2)));
typedef float    v4f   __attribute__((ext_vector_type(4)));
typedef unsigned u32x4 __attribute__((ext_vector_type(4)));
typedef unsigned u32x3 __attribute__((ext_vector_type(3)));
#define LDS_AS __attribute__((address_space(3)))

constexpr int NR = 8;
constexpr int KD = 4096;
constexpr int N_ALL = 16384;
constexpr int N_LOC = 2048;
constexpr int M_SH = 512;
constexpr int NWG = 256;
constexpr int NTHR = 512;
constexpr int NKS = 32;
constexpr int XT_B = 16384;
constexpr int WT_B = 32768;
constexpr int LDS_X = 0;
constexpr int LDS_CTL = 3 * XT_B;
constexpr int LDS_W = 4 * XT_B;
constexpr int LDS_BYTES = LDS_W + 3 * WT_B;

constexpr int GRP_P = 1536, GRP_E = 112, ECAP = 52;
constexpr int TREC = 8 * (GRP_P + GRP_E);
constexpr unsigned RAW_FLAG = 0x80u;
constexpr int SCR_B = 4608;

constexpr size_t WS_CNT  = 0;
constexpr size_t WS_TMO  = WS_CNT + NWG * 128;
constexpr size_t WS_LEAD = WS_TMO + 128;
constexpr size_t WS_INV  = WS_LEAD + 16 * 128;
constexpr size_t WS_FLAG = WS_INV + 16 * 128 + 3968;
constexpr size_t WS_X    = 262144;
constexpr size_t RAW_OFF = 1703936;
constexpr size_t SLOT_B  = RAW_OFF + (size_t)M_SH * KD;
constexpr size_t WS_NEED = WS_X + 2 * SLOT_B;
static_assert(WS_FLAG % 128 == 0 && WS_FLAG + NR * NWG * 64 <= WS_X, "ws layout");
static_assert(4 * NKS * TREC <= RAW_OFF, "ws layout");

constexpr unsigned long long DEADLINE_TICKS = 30000000ull;

struct Params {
  const uint8_t* x; const uint8_t* w; const float* sx; const float* sw; float* out;
  uint8_t* ws[NR];
  int rank; int pad;
};

#define STAMP(i) do {} while (0)
#define WAITV(n) asm volatile("s_waitcnt vmcnt(%0)" :: "n"(n) : "memory")
#define BARRIER() asm volatile("s_waitcnt lgkmcnt(0)\n\ts_barrier" ::: "memory")
#define RAW_BARRIER() asm volatile("s_barrier" ::: "memory")
#define SCHED() __builtin_amdgcn_sched_barrier(0)

__device__ __forceinline__ void st16(void* p, u32x4 v) { asm volatile("global_store_dwordx4 %0, %1, off sc0 sc1\n\ts_nop 1" :: "v"(p), "v"(v) : "memory"); }
__device__ __forceinline__ void st12(void* p, u32x3 v) { asm volatile("global_store_dwordx3 %0, %1, off sc0 sc1\n\ts_nop 1" :: "v"(p), "v"(v) : "memory"); }
__device__ __forceinline__ void st4(void* p, unsigned v) { asm volatile("global_store_dword %0, %1, off sc0 sc1" :: "v"(p), "v"(v) : "memory"); }
__device__ __forceinline__ void store_out16(void* p, v4f v) { asm volatile("global_store_dwordx4 %0, %1, off sc0 sc1\n\ts_nop 1" :: "v"(p), "v"(v) : "memory"); }

__device__ __forceinline__ unsigned wave_scan(unsigned v) {
  v += __builtin_amdgcn_update_dpp(0u, v, 0x111, 0xf, 0xf, true);
  v += __builtin_amdgcn_update_dpp(0u, v, 0x112, 0xf, 0xf, true);
  v += __builtin_amdgcn_update_dpp(0u, v, 0x114, 0xf, 0xf, true);
  v += __builtin_amdgcn_update_dpp(0u, v, 0x118, 0xf, 0xf, true);
  v += __builtin_amdgcn_update_dpp(0u, v, 0x142, 0xa, 0xf, true);
  v += __builtin_amdgcn_update_dpp(0u, v, 0x143, 0xc, 0xf, true);
  return v;
}

__device__ __forceinline__ u32x4 make_srd(const void* base, unsigned bytes) {
  const unsigned long long a = (unsigned long long)base;
  return (u32x4){(unsigned)__builtin_amdgcn_readfirstlane((unsigned)a), (unsigned)__builtin_amdgcn_readfirstlane((unsigned)(a >> 32)) & 0xFFFFu,
                 (unsigned)__builtin_amdgcn_readfirstlane(bytes), 0x00020000u};
}

__device__ __forceinline__ bool wait_words(const uint8_t* addr, unsigned call, unsigned* tmo, int lane) {
  const unsigned long long t0 = __builtin_amdgcn_s_memrealtime();
  while (true) {
    unsigned v, t;
    asm volatile("global_load_dword %0, %2, off sc0 sc1\n\tglobal_load_dword %1, %3, off sc1\n\ts_waitcnt vmcnt(0)"
                 : "=&v"(v), "=&v"(t) : "v"(addr), "v"(tmo) : "memory");
    if (__all(v >= call)) return true;
    if ((unsigned)__builtin_amdgcn_readfirstlane(t) == call || __builtin_amdgcn_s_memrealtime() - t0 > DEADLINE_TICKS) {
      if (lane == 0) __hip_atomic_store(tmo, call, __ATOMIC_RELAXED, __HIP_MEMORY_SCOPE_AGENT);
      return false;
    }
  }
}

struct XRegs { u32x3 p[4]; unsigned e; };

__global__ __launch_bounds__(NTHR, 2) void gemm_ag_kernel(Params p) {
  extern __shared__ __attribute__((aligned(16))) uint8_t smem_[];
  LDS_AS uint8_t* smem = (LDS_AS uint8_t*)smem_;
  const int tid = threadIdx.x, lane = tid & 63;
  const int wid = __builtin_amdgcn_readfirstlane(tid >> 6);
  const int b = blockIdx.x;
  uint8_t* const ws = p.ws[p.rank];
  if (wid == 0) STAMP(0);

  unsigned* cnt = (unsigned*)(ws + WS_CNT + (size_t)b * 128);
  const int G = b * 4 + (wid >> 1), half = wid & 1;
  const uint8_t* gx = p.x + (size_t)((G >> 8) * 128 + (G & 7) * 16 + (lane & 15)) * KD + ((G >> 3) & 31) * 128 + half * 64 + (lane >> 4) * 16;
  unsigned call_raw; u32x4 d;
  asm volatile("global_load_dwordx4 %0, %2, off\n\tglobal_load_dword %1, %3, off sc1" : "=&v"(d), "=&v"(call_raw) : "v"(gx), "v"(cnt) : "memory");

  const int grp = b & 7, idx = b >> 3;
  const int src = (p.rank + grp) & (NR - 1);
  const int rb = idx >> 3, cb = idx & 7;
  const int col0 = p.rank * N_LOC + cb * 256;

  unsigned long long peer = (unsigned long long)p.ws[0];
#pragma unroll
  for (int i = 1; i < NR; i++) {
    unsigned long long wi = (unsigned long long)p.ws[i];
    asm volatile("" : "+s"(wi));
    peer = (lane == i) ? wi : peer;
  }
  uint8_t* const myflag = (uint8_t*)peer + WS_FLAG + ((size_t)p.rank * NWG + b) * 64;

  LDS_AS uint8_t* sc = smem + LDS_X + (wid >> 1) * SCR_B + 2048;
  if (half == 0 && lane < 28) ((LDS_AS unsigned*)sc)[lane] = (unsigned)(lane * 8) | ((unsigned)(lane * 8 + 4) << 16);
  BARRIER();
  asm volatile("s_waitcnt vmcnt(0)" : "+v"(d), "+v"(call_raw) :: "memory");
  const unsigned call = __builtin_amdgcn_readfirstlane(call_raw) + 1u;
  const size_t slot = WS_X + (size_t)(call & 1u) * SLOT_B;
  const unsigned xcc = (unsigned)__builtin_amdgcn_s_getreg((3 << 11) | 20) & 0xFu;
  unsigned* const tmo = (unsigned*)(ws + WS_TMO);
  unsigned lead_old = call;
  if (wid == 5 && lane == 0)
    asm volatile("global_atomic_umax %0, %1, %2, off sc0" : "+v"(lead_old) : "v"(ws + WS_LEAD + xcc * 128), "v"(call) : "memory");
  {
    const unsigned esum = (unsigned)__builtin_amdgcn_readlane(wave_scan(
        __builtin_amdgcn_udot4((d[0] >> 2) & 0x1F1F1F1Fu, 0x01010101u,
        __builtin_amdgcn_udot4((d[1] >> 2) & 0x1F1F1F1Fu, 0x01010101u,
        __builtin_amdgcn_udot4((d[2] >> 2) & 0x1F1F1F1Fu, 0x01010101u,
        __builtin_amdgcn_udot4((d[3] >> 2) & 0x1F1F1F1Fu, 0x01010101u, 0u, false), false), false), false)), 63);
    const unsigned base = esum < 4280u ? 0u : ((esum - 4280u) >> 10 > 24u ? 24u : (esum - 4280u) >> 10);
    const unsigned b4 = base * 0x04040404u;
    unsigned c[4], P[4];
#pragma unroll
    for (int i = 0; i < 4; i++) {
      const unsigned t = (d[i] | 0x80808080u) - b4;
      c[i] = (d[i] & 0x80808080u) | (t & 0x1F1F1F1Fu);
      P[i] = (c[i] + b4) ^ d[i];
    }
    const u32x3 o = {c[0] | ((c[3] & 0x03030303u) << 5), c[1] | ((c[3] & 0x0C0C0C0Cu) << 3),
                     c[2] | ((c[3] & 0x10101010u) << 1) | ((c[3] & 0x80808080u) >> 1)};
    uint8_t* trec = ws + slot + (size_t)(G >> 3) * TREC;
    st12(trec + (G & 7) * GRP_P + half * 768 + lane * 12, o);
    const unsigned z0 = (P[0] + 0x7C7C7C7Cu) & 0x80808080u, z1 = (P[1] + 0x7C7C7C7Cu) & 0x80808080u;
    const unsigned z2 = (P[2] + 0x7C7C7C7Cu) & 0x80808080u, z3 = (P[3] + 0x7C7C7C7Cu) & 0x80808080u;
    unsigned mask = (__builtin_amdgcn_udot4(z0, 0x08040201u, __builtin_amdgcn_udot4(z1, 0x80402010u, 0u, false), false) >> 7) |
                    (__builtin_amdgcn_udot4(z2, 0x08040201u, __builtin_amdgcn_udot4(z3, 0x80402010u, 0u, false), false) << 1);
    const unsigned ncnt = (unsigned)__builtin_popcount(mask);
    const unsigned incl = wave_scan(ncnt);
    const unsigned ne = (unsigned)__builtin_amdgcn_readlane(incl, 63);
    const int dir = half ? -2 : 2;
    LDS_AS uint8_t* ea = sc + (half ? 110 : 8) + (int)(incl - ncnt) * dir;
    const unsigned long long p01 = ((unsigned long long)P[1] << 32) | P[0], p23 = ((unsigned long long)P[3] << 32) | P[2];
    const unsigned pos0 = ((unsigned)half << 10) | ((unsigned)lane << 4);
    while (mask) {
      const unsigned j = (unsigned)__builtin_ctz(mask);
      const unsigned patch = (unsigned)((j & 8u ? p23 : p01) >> ((j & 7u) * 8u + 2u)) & 31u;
      *(LDS_AS unsigned short*)ea = (unsigned short)(pos0 | j | (patch << 11));
      ea += dir;
      mask &= mask - 1u;
    }
    if (lane == 0) *(LDS_AS unsigned*)(sc + 2304 + half * 4) = ne | (base << 16);
    if (wid == 0) STAMP(101);
    BARRIER();
    const unsigned m0 = *(LDS_AS unsigned*)(sc + 2304), m1 = *(LDS_AS unsigned*)(sc + 2308);
    const bool raw = (m0 & 0xFFFFu) + (m1 & 0xFFFFu) > (unsigned)ECAP;
    if (raw) st16(ws + slot + RAW_OFF + (size_t)(G * 2 + half) * 1024 + lane * 16, d);
    if (half == 0 && lane < 28) {
      unsigned v = ((LDS_AS unsigned*)sc)[lane];
      if (lane == 0) v = (m0 >> 16) * 0x04040404u | (raw ? RAW_FLAG : 0u);
      if (lane == 1) v = (m1 >> 16) * 0x04040404u;
      st4(trec + 8 * GRP_P + (G & 7) * GRP_E + lane * 4, v);
    }
    asm volatile("s_waitcnt vmcnt(0)" : "+v"(lead_old) :: "memory");
    BARRIER();
    if (wid == 0) {
      STAMP(1);
      if (lane < NR) st4(myflag, call);
      if (lane == 0) __hip_atomic_store(cnt, call, __ATOMIC_RELAXED, __HIP_MEMORY_SCOPE_AGENT);
    }
    if (wid == 5 && __builtin_amdgcn_readfirstlane(lead_old) < call) {
      __builtin_amdgcn_fence(__ATOMIC_ACQUIRE, "");
      WAITV(0);
      if (lane == 0) __hip_atomic_store((unsigned*)(ws + WS_INV + xcc * 128), call, __ATOMIC_RELAXED, __HIP_MEMORY_SCOPE_AGENT);
      STAMP(102);
    }
  }

  const int pj = wid & 3;
  unsigned wv[8];
#pragma unroll
  for (int i = 0; i < 8; i++)
    wv[i] = (unsigned)(lane >> 4) * N_ALL + (unsigned)(((lane & 15) ^ ((lane >> 4) | ((i & 1) << 2) | (((i >> 2) & 1) << 3))) * 16) +
            (unsigned)(i * 4 * N_ALL - (i & 3) * 1024);
  const u32x4 wsrd = make_srd(p.w + (size_t)pj * 32 * N_ALL + col0, (unsigned)((size_t)KD * N_ALL - (size_t)pj * 32 * N_ALL - col0));
  const unsigned wm0 = (unsigned)(LDS_W + pj * 8192);
#define W_FIRST(i, so, slotoff) asm volatile("s_add_u32 m0, %3, %4\n\ts_nop 0\n\tbuffer_load_dwordx4 %0, %1, %2 offen lds" \
      :: "v"(wv[i]), "s"(wsrd), "s"(so), "s"(wm0), "n"(slotoff) : "memory", "scc")
#define W_NEXT(i, so) asm volatile("buffer_load_dwordx4 %0, %1, %2 offen offset:%3 lds" :: "v"(wv[i]), "s"(wsrd), "s"(so), "n"(((i) & 3) * 1024) : "memory")
  if (wid >= 4) {
    const unsigned so = 0u;
    W_FIRST(0, so, 0); W_NEXT(1, so); W_NEXT(2, so); W_NEXT(3, so); W_FIRST(4, so, 4096); W_NEXT(5, so); W_NEXT(6, so); W_NEXT(7, so);
  }

  volatile LDS_AS unsigned* ctl = (volatile LDS_AS unsigned*)(smem + LDS_CTL);
  const uint8_t* flagp = ws + WS_FLAG + ((size_t)src * NWG + rb * 64 + lane) * 64;
  if (wid == 4) {
    STAMP(105);
    const bool ok = wait_words(lane < 8 ? flagp : ws + WS_INV + xcc * 128, call, tmo, lane);
    STAMP(2);
    if (lane == 0) ctl[0] = ok ? 0u : 1u;
  }
  BARRIER();
  const bool bad = __builtin_amdgcn_readfirstlane(ctl[0]) != 0u;

  const int wn = wid & 3;
  float* const outp = p.out + (size_t)(src * M_SH + rb * 128 + (lane & 15)) * N_LOC + cb * 256 + wn * 64 + (lane >> 4) * 4;
  auto store_nan = [&]() {
    const float n = __builtin_nanf("");
    const v4f nv = {n, n, n, n};
    for (int mf = 0; mf < 8; mf++)
      for (int nf = 0; nf < 4; nf++) store_out16(outp + (size_t)(mf * 16) * N_LOC + nf * 16, nv);
  };
  if (bad) {
    WAITV(0);
    if (wid < 4) store_nan();
    return;
  }

  if (wid >= 4) {
    const uint8_t* const xsrc = p.ws[src] + slot;
    u32x4 xsrd = make_srd(xsrc + (size_t)rb * NKS * TREC + pj * 2 * GRP_P, (unsigned)(4 * NKS * TREC - rb * NKS * TREC - pj * 2 * GRP_P));
    const unsigned xv = (unsigned)lane * 12u;
    const int el = lane & 31;
    const unsigned ev = (unsigned)(8 * GRP_P + (2 * pj + (lane >> 5)) * GRP_E + (el < 28 ? el : 27) * 4 - pj * 2 * GRP_P);
    const unsigned emask = (el < 2 || el >= 28) ? 0u : 0xFFFFFFFFu;
    const unsigned edef = (el < 2 || el >= 28) ? (unsigned)(el * 8) | ((unsigned)(el * 8 + 4) << 16) : 0u;
    unsigned xw = (unsigned)(LDS_X + pj * 4096 + lane * 16);
    unsigned xg = (unsigned)(LDS_X + (2 * pj + (lane >> 5)) * 2048);
    asm volatile("" : "+v"(xw), "+v"(xg));
    XRegs X[3];
#define X_LD(r, j, so) asm volatile("buffer_load_dwordx3 %0, %1, %2, %3 offen offset:%4" : "=v"((r).p[j]) : "v"(xv), "s"(xsrd), "s"(so), "n"((j) * 768) : "memory")
#define X_LE(r, so) asm volatile("buffer_load_dword %0, %1, %2, %3 offen" : "=v"((r).e) : "v"(ev), "s"(xsrd), "s"(so) : "memory")
#define X_ALL(r, u) do { const unsigned so_ = (unsigned)(u) * (unsigned)TREC; X_LD(r, 0, so_); X_LD(r, 1, so_); X_LD(r, 2, so_); X_LD(r, 3, so_); X_LE(r, so_); } while (0)
    auto raw_fix = [&](int u, unsigned hA0, unsigned hB0, LDS_AS uint8_t* dst) {
      if (xsrd[2] == 0u) return;
      const uint8_t* g = xsrc + RAW_OFF + ((size_t)(rb * NKS + u) * 16 + pj * 4) * 1024;
#pragma unroll
      for (int k = 0; k < 2; k++) {
        if ((k ? hB0 : hA0) & RAW_FLAG) {
          u32x4 a, c;
          asm volatile("global_load_dwordx4 %0, %2, %3\n\t"
                       "global_load_dwordx4 %1, %2, %3 offset:1024\n\t"
                       "s_waitcnt vmcnt(0)"
                       : "=&v"(a), "=&v"(c) : "v"((unsigned)lane * 16u), "s"(g + k * 2048) : "memory");
          *(LDS_AS u32x4*)(dst + k * 2048) = a;
          *(LDS_AS u32x4*)(dst + k * 2048 + 1024) = c;
        }
      }
    };
    X_ALL(X[0], 0); X_ALL(X[1], 1);
#define PT(x) do {} while (0)
#define PACC() do {} while (0)
#define UNP_LO(o, b) (u32x3){((o)[0] & 0x9F9F9F9Fu) + (b), ((o)[1] & 0x9F9F9F9Fu) + (b), ((o)[2] & 0x9F9F9F9Fu) + (b)}
#define UNP_HI(o, b) ((((o)[0] >> 5) & 0x03030303u) | (((o)[1] >> 3) & 0x0C0C0C0Cu) | (((o)[2] >> 1) & 0x10101010u) | (((o)[2] << 1) & 0x80808080u)) + (b)
#define PACK(j, b) do { \
      const unsigned h_ = UNP_HI(r.p[j], b); SCHED(); \
      const u32x3 l_ = UNP_LO(r.p[j], b); \
      *(LDS_AS u32x4*)(dst + (j) * 1024) = (u32x4){l_[0], l_[1], l_[2], h_}; SCHED(); } while (0)
#define W_I(i, so, off) do { if ((i) == 0) W_FIRST(0, so, off); else if ((i) == 4) W_FIRST(4, so, (off) + 4096); else W_NEXT(i, so); } while (0)
#define PROD(v, k, DO_W, DO_X, NW) do { \
      XRegs& r = X[(k)]; XRegs& rn = X[((k) + 2) % 3]; \
      const unsigned soW = (unsigned)((v) + 1) * (128u * N_ALL), soX = (unsigned)((v) + 2) * (unsigned)TREC; \
      if (DO_W) { W_I(0, soW, (((k) + 1) % 3) * WT_B); W_I(1, soW, 0); W_I(2, soW, 0); W_I(3, soW, 0); W_I(4, soW, (((k) + 1) % 3) * WT_B); W_I(5, soW, 0); W_I(6, soW, 0); W_I(7, soW, 0); } \
      if (DO_X) { X_LD(rn, 0, soX); X_LD(rn, 1, soX); X_LD(rn, 2, soX); X_LD(rn, 3, soX); X_LE(rn, soX); } \
      PT(pt0); NW; PT(pt2); \
      asm volatile("" : "+v"(r.p[0]), "+v"(r.p[1]), "+v"(r.p[2]), "+v"(r.p[3]), "+v"(r.e));       \
      const unsigned hA0 = (unsigned)__builtin_amdgcn_readlane(r.e, 0), hA1 = (unsigned)__builtin_amdgcn_readlane(r.e, 1); \
      const unsigned hB0 = (unsigned)__builtin_amdgcn_readlane(r.e, 32), hB1 = (unsigned)__builtin_amdgcn_readlane(r.e, 33); \
      LDS_AS uint8_t* dst = (LDS_AS uint8_t*)(size_t)xw + (k) * XT_B; \
      PACK(0, hA0); PACK(1, hA1); PACK(2, hB0); \
      const unsigned e = (r.e & emask) | edef; \
      PACK(3, hB1); \
        \
      const unsigned a0 = (e & 0x7FCu) | xg, a1 = ((e >> 16) & 0x7FCu) | xg; \
      const unsigned v0_ = ((e >> 11) & 31u) << (((e << 3) & 24u) | 2u); SCHED(); \
      const unsigned v1_ = (e >> 27) << (((e >> 13) & 24u) | 2u); \
      __hip_atomic_fetch_xor((LDS_AS unsigned*)((LDS_AS uint8_t*)(size_t)a0 + (k) * XT_B), v0_, __ATOMIC_RELAXED, __HIP_MEMORY_SCOPE_WORKGROUP); \
      __hip_atomic_fetch_xor((LDS_AS unsigned*)((LDS_AS uint8_t*)(size_t)a1 + (k) * XT_B), v1_, __ATOMIC_RELAXED, __HIP_MEMORY_SCOPE_WORKGROUP); \
      SCHED(); \
      if ((hA0 | hB0) & RAW_FLAG) raw_fix((v), hA0, hB0, dst); \
      PT(pt3); \
      BARRIER(); PT(pt4); PACC(); } while (0)
    PROD(0, 0, true, true, WAITV(18));
    PROD(1, 1, true, true, WAITV(18));
    if (__builtin_amdgcn_readfirstlane(ctl[1]) != 0u) xsrd[2] = 0u;
    PROD(2, 2, true, true, WAITV(18));
    for (int v0 = 3; v0 < 30; v0 += 3) {
      PROD(v0 + 0, 0, true, true, WAITV(18));
      PROD(v0 + 1, 1, true, true, WAITV(18));
      PROD(v0 + 2, 2, true, true, WAITV(18));
    }
    PROD(30, 0, true, false, WAITV(13));
    PROD(31, 1, false, false, WAITV(0));
    return;
  }

  const float scale = p.sx[0] * p.sw[0];
  const unsigned xaddr = (unsigned)(LDS_X + lane * 16);
  unsigned waddr[4];
  {
    const int g = lane >> 4, q = (lane & 15) >> 1;
    const int f = q | ((g & 1) << 3);
#pragma unroll
    for (int nf = 0; nf < 4; nf++)
      waddr[nf] = (unsigned)(LDS_W + (16 * g + q) * 256 + (((wn * 4 + nf) ^ f) << 4) + (lane & 1) * 8);
  }
  v4f acc[4][8];
#pragma unroll
  for (int i = 0; i < 4; i++)
#pragma unroll
    for (int j = 0; j < 8; j++) acc[i][j] = (v4f){0.f, 0.f, 0.f, 0.f};
#define READ_W(wf, nf, off) do { LDS_AS uint8_t* wp_ = smem + waddr[nf] + (off); \
    const v2i t0_ = __builtin_amdgcn_ds_read_tr8_b64_v2i32((LDS_AS v2i*)(wp_)); \
    const v2i t1_ = __builtin_amdgcn_ds_read_tr8_b64_v2i32((LDS_AS v2i*)(wp_ + 2048)); \
    const v2i t2_ = __builtin_amdgcn_ds_read_tr8_b64_v2i32((LDS_AS v2i*)(wp_ + 16384)); \
    const v2i t3_ = __builtin_amdgcn_ds_read_tr8_b64_v2i32((LDS_AS v2i*)(wp_ + 18432)); \
    wf[nf] = (v8i){t0_[0], t0_[1], t1_[0], t1_[1], t2_[0], t2_[1], t3_[0], t3_[1]}; } while (0)
#define READ_X(r, mf, off) do { LDS_AS uint8_t* xp_ = smem + xaddr + (off) + (mf) * 2048; \
    const v4i a_ = *(LDS_AS v4i*)(xp_); const v4i c_ = *(LDS_AS v4i*)(xp_ + 1024); \
    xf[r] = (v8i){a_[0], a_[1], a_[2], a_[3], c_[0], c_[1], c_[2], c_[3]}; } while (0)
#define CBARRIER(n) asm volatile("s_waitcnt lgkmcnt(%0)\n\ts_barrier" :: "n"(n) : "memory")
  if (wid == 0) {
    const bool ok = wait_words(flagp, call, tmo, lane);
    if (lane == 0) ctl[1] = ok ? 0u : 1u;
  }
  unsigned tmo_late;
  v8i wf[4], xf[8];
#define MM(nf, mf) acc[nf][mf] = __builtin_amdgcn_mfma_scale_f32_16x16x128_f8f6f4(wf[nf], xf[mf], acc[nf][mf], 1, 1, 0, 0, 0, 0)
#define MROW_NR(mf) do { MM(0, mf); MM(1, mf); MM(2, mf); MM(3, mf); } while (0)
#define MROW(mf, xo) do { MROW_NR(mf); SCHED(); READ_X(mf, mf, xo); SCHED(); } while (0)
#define STEP_NB(xo, wo) do { \
    MROW(0, xo); MROW(1, xo); MROW(2, xo); MROW(3, xo); MROW(4, xo); MROW(5, xo); MROW(6, xo); \
    MM(0, 7); SCHED(); READ_W(wf, 0, wo); SCHED(); MM(1, 7); SCHED(); READ_W(wf, 1, wo); SCHED(); \
    MM(2, 7); SCHED(); READ_W(wf, 2, wo); SCHED(); MM(3, 7); SCHED(); READ_W(wf, 3, wo); READ_X(7, 7, xo); SCHED(); } while (0)
  RAW_BARRIER();
  if (wid == 0) STAMP(4);
#pragma unroll
  for (int i = 0; i < 4; i++) READ_W(wf, i, 0);
#pragma unroll
  for (int i = 0; i < 8; i++) READ_X(i, i, 0);
  BARRIER();
  asm volatile("global_load_dword %0, %1, off sc1" : "=v"(tmo_late) : "v"(tmo) : "memory");
  int sl = 1;
  for (int u = 0; u < NKS - 3; u++) {
    STEP_NB(sl * XT_B, sl * WT_B); CBARRIER(0);
    sl = (sl + 1 == 3) ? 0 : sl + 1;
    if (wid == 0 && (u % 6) == 0) STAMP(8 + u);
  }
  asm volatile("s_waitcnt vmcnt(0)" : "+v"(tmo_late) :: "memory");
  if (__builtin_amdgcn_readfirstlane(tmo_late) == call) { store_nan(); return; }
  {
    STEP_NB(((NKS - 2) % 3) * XT_B, ((NKS - 2) % 3) * WT_B);
    constexpr int xo = ((NKS - 1) % 3) * XT_B, wo = ((NKS - 1) % 3) * WT_B;
    SCHED();
    MROW_NR(0); MROW_NR(1); MROW_NR(2); MROW_NR(3); MROW_NR(4); MROW_NR(5); MROW_NR(6); MROW_NR(7);
    SCHED();
    RAW_BARRIER();
    READ_W(wf, 0, wo); READ_X(0, 0, xo); READ_W(wf, 1, wo); READ_W(wf, 2, wo); READ_W(wf, 3, wo); READ_X(1, 1, xo); SCHED();
#define EPILOGUE(mf) do { _Pragma("unroll") for (int nf = 0; nf < 4; nf++) { \
      v4f y = acc[nf][mf] * scale; \
      y[0] = fmaxf(y[0], 0.f); y[1] = fmaxf(y[1], 0.f); y[2] = fmaxf(y[2], 0.f); y[3] = fmaxf(y[3], 0.f); \
      store_out16(outp + (size_t)((mf) * 16) * N_LOC + nf * 16, y); } SCHED(); } while (0)
    MROW_NR(0); SCHED(); READ_X(2, 2, xo); SCHED(); MROW_NR(1); SCHED(); READ_X(3, 3, xo); SCHED(); EPILOGUE(0);
    MROW_NR(2); SCHED(); READ_X(4, 4, xo); SCHED(); EPILOGUE(1); MROW_NR(3); SCHED(); READ_X(5, 5, xo); SCHED(); EPILOGUE(2);
    MROW_NR(4); SCHED(); READ_X(6, 6, xo); SCHED(); EPILOGUE(3); MROW_NR(5); SCHED(); READ_X(7, 7, xo); SCHED(); EPILOGUE(4);
    MROW_NR(6); SCHED(); EPILOGUE(5); MROW_NR(7); SCHED(); EPILOGUE(6); EPILOGUE(7);
  }
}

}

extern "C" void kernel_launch(int rank, int nranks, ncclComm_t comm,
                              hipStream_t stream,
                              void* const* d_in, const int* in_sizes, int n_in,
                              void* d_out, int out_size,
                              const int* in_dtypes, int out_dtype,
                              void* const* d_out_all,
                              void* const* d_ws_all, size_t ws_bytes) {
  (void)comm; (void)in_sizes; (void)in_dtypes; (void)out_dtype; (void)d_out_all; (void)out_size;
  if (nranks != NR || n_in != 4 || ws_bytes < WS_NEED || (unsigned)rank >= (unsigned)NR) {
    fprintf(stderr, "kernel_launch: unexpected configuration\n");
    return;
  }
  static bool init = false;
  if (!init) {
    HIP_CHECK(hipFuncSetAttribute((const void*)gemm_ag_kernel, hipFuncAttributeMaxDynamicSharedMemorySize, LDS_BYTES));
    init = true;
  }
  Params p;
  p.x = (const uint8_t*)d_in[0]; p.w = (const uint8_t*)d_in[1];
  p.sx = (const float*)d_in[2]; p.sw = (const float*)d_in[3];
  p.out = (float*)d_out;
  for (int i = 0; i < NR; i++) p.ws[i] = (uint8_t*)d_ws_all[i];
  p.rank = rank; p.pad = 0;
  gemm_ag_kernel<<<NWG, NTHR, LDS_BYTES, stream>>>(p);
  HIP_CHECK(hipGetLastError());
}
```

```cpp
#include <hip/hip_runtime.h>
#include <rccl/rccl.h>
#include <cstdio>
#include <cstdint>

#define HIP_CHECK(expr) do { hipError_t _e = (expr); if (_e != hipSuccess) { \
    fprintf(stderr, "HIP error %s at %s:%d\n", hipGetErrorString(_e), __FILE__, __LINE__); \
    return; } } while (0)

namespace {

typedef int      v8i   __attribute__((ext_vector_type(8)));
typedef int      v4i   __attribute__((ext_vector_type(4)));
typedef int      v2i   __attribute__((ext_vector_type(2)));
typedef float    v4f   __attribute__((ext_vector_type(4)));
typedef unsigned u32x4 __attribute__((ext_vector_type(4)));
typedef unsigned u32x3 __attribute__((ext_vector_type(3)));
#define LDS_AS __attribute__((address_space(3)))

constexpr int NR = 8;
constexpr int KD = 4096;
constexpr int N_ALL = 16384;
constexpr int N_LOC = 2048;
constexpr int M_SH = 512;
constexpr int NWG = 256;
constexpr int NTHR = 512;
constexpr int NKS = 32;
constexpr int XT_B = 16384;
constexpr int WT_B = 32768;
constexpr int LDS_X = 0;
constexpr int LDS_W = 2 * XT_B;
constexpr int LDS_BYTES = LDS_W + 4 * WT_B;
constexpr int LDS_CTL = LDS_W + 3 * WT_B;

constexpr int GRP_P = 1536, GRP_E = 112, ECAP = 52;
constexpr int TREC = 8 * (GRP_P + GRP_E);
constexpr unsigned RAW_FLAG = 0x80u;
constexpr int SCR_B = 4608;

constexpr size_t WS_CNT  = 0;
constexpr size_t WS_TMO  = WS_CNT + NWG * 128;
constexpr size_t WS_LEAD = WS_TMO + 128;
constexpr size_t WS_INV  = WS_LEAD + 16 * 128;
constexpr size_t WS_FLAG = WS_INV + 16 * 128 + 3968;
constexpr size_t WS_X    = 262144;
constexpr size_t RAW_OFF = 1703936;
constexpr size_t SLOT_B  = RAW_OFF + (size_t)M_SH * KD;
constexpr size_t WS_NEED = WS_X + 2 * SLOT_B;
static_assert(WS_FLAG % 128 == 0 && WS_FLAG + NR * NWG * 64 <= WS_X, "ws layout");
static_assert(4 * NKS * TREC <= RAW_OFF, "ws layout");

constexpr unsigned long long DEADLINE_TICKS = 30000000ull;

struct Params {
  const uint8_t* x; const uint8_t* w; const float* sx; const float* sw; float* out;
  uint8_t* ws[NR];
  int rank; int pad;
};

#define STAMP(i) do {} while (0)
#define WAITV(n) asm volatile("s_waitcnt vmcnt(%0)" :: "n"(n) : "memory")
#define BARRIER() asm volatile("s_waitcnt lgkmcnt(0)\n\ts_barrier" ::: "memory")
#define RAW_BARRIER() asm volatile("s_barrier" ::: "memory")
#define SCHED() __builtin_amdgcn_sched_barrier(0)

__device__ __forceinline__ void st16(void* p, u32x4 v) { asm volatile("global_store_dwordx4 %0, %1, off sc0 sc1\n\ts_nop 1" :: "v"(p), "v"(v) : "memory"); }
__device__ __forceinline__ void st12(void* p, u32x3 v) { asm volatile("global_store_dwordx3 %0, %1, off sc0 sc1\n\ts_nop 1" :: "v"(p), "v"(v) : "memory"); }
__device__ __forceinline__ void st4(void* p, unsigned v) { asm volatile("global_store_dword %0, %1, off sc0 sc1" :: "v"(p), "v"(v) : "memory"); }
__device__ __forceinline__ void store_out16(void* p, v4f v) { asm volatile("global_store_dwordx4 %0, %1, off sc0 sc1\n\ts_nop 1" :: "v"(p), "v"(v) : "memory"); }

__device__ __forceinline__ unsigned wave_scan(unsigned v) {
  v += __builtin_amdgcn_update_dpp(0u, v, 0x111, 0xf, 0xf, true);
  v += __builtin_amdgcn_update_dpp(0u, v, 0x112, 0xf, 0xf, true);
  v += __builtin_amdgcn_update_dpp(0u, v, 0x114, 0xf, 0xf, true);
  v += __builtin_amdgcn_update_dpp(0u, v, 0x118, 0xf, 0xf, true);
  v += __builtin_amdgcn_update_dpp(0u, v, 0x142, 0xa, 0xf, true);
  v += __builtin_amdgcn_update_dpp(0u, v, 0x143, 0xc, 0xf, true);
  return v;
}

__device__ __forceinline__ u32x4 make_srd(const void* base, unsigned bytes) {
  const unsigned long long a = (unsigned long long)base;
  return (u32x4){(unsigned)__builtin_amdgcn_readfirstlane((unsigned)a), (unsigned)__builtin_amdgcn_readfirstlane((unsigned)(a >> 32)) & 0xFFFFu,
                 (unsigned)__builtin_amdgcn_readfirstlane(bytes), 0x00020000u};
}

__device__ __forceinline__ bool wait_words(const uint8_t* addr, unsigned call, unsigned* tmo, int lane) {
  const unsigned long long t0 = __builtin_amdgcn_s_memrealtime();
  while (true) {
    unsigned v, t;
    asm volatile("global_load_dword %0, %2, off sc0 sc1\n\tglobal_load_dword %1, %3, off sc1\n\ts_waitcnt vmcnt(0)"
                 : "=&v"(v), "=&v"(t) : "v"(addr), "v"(tmo) : "memory");
    if (__all(v >= call)) return true;
    if ((unsigned)__builtin_amdgcn_readfirstlane(t) == call || __builtin_amdgcn_s_memrealtime() - t0 > DEADLINE_TICKS) {
      if (lane == 0) __hip_atomic_store(tmo, call, __ATOMIC_RELAXED, __HIP_MEMORY_SCOPE_AGENT);
      return false;
    }
  }
}

struct XRegs { u32x3 p[4]; unsigned e; };

__global__ __launch_bounds__(NTHR, 2) void gemm_ag_kernel(Params p) {
  extern __shared__ __attribute__((aligned(16))) uint8_t smem_[];
  LDS_AS uint8_t* smem = (LDS_AS uint8_t*)smem_;
  const int tid = threadIdx.x, lane = tid & 63;
  const int wid = __builtin_amdgcn_readfirstlane(tid >> 6);
  const int b = blockIdx.x;
  uint8_t* const ws = p.ws[p.rank];
  if (wid == 0) STAMP(0);

  unsigned* cnt = (unsigned*)(ws + WS_CNT + (size_t)b * 128);
  const int G = b * 4 + (wid >> 1), half = wid & 1;
  const uint8_t* gx = p.x + (size_t)((G >> 8) * 128 + (G & 7) * 16 + (lane & 15)) * KD + ((G >> 3) & 31) * 128 + half * 64 + (lane >> 4) * 16;
  unsigned call_raw; u32x4 d;
  asm volatile("global_load_dwordx4 %0, %2, off\n\tglobal_load_dword %1, %3, off sc1" : "=&v"(d), "=&v"(call_raw) : "v"(gx), "v"(cnt) : "memory");

  const int grp = b & 7, idx = b >> 3;
  const int src = (p.rank + grp) & (NR - 1);
  const int rb = idx >> 3, cb = idx & 7;
  const int col0 = p.rank * N_LOC + cb * 256;
#define XD 2
#define XPOS 0
#define KROT 4
  const unsigned krot = (unsigned)(grp * KROT);
#define KST(u) (((unsigned)(u) + krot) & 31u)

  unsigned long long peer = (unsigned long long)p.ws[0];
#pragma unroll
  for (int i = 1; i < NR; i++) {
    unsigned long long wi = (unsigned long long)p.ws[i];
    asm volatile("" : "+s"(wi));
    peer = (lane == i) ? wi : peer;
  }
  uint8_t* const myflag = (uint8_t*)peer + WS_FLAG + ((size_t)p.rank * NWG + b) * 64;

  LDS_AS uint8_t* sc = smem + LDS_X + (wid >> 1) * SCR_B + 2048;
  if (half == 0 && lane < 28) ((LDS_AS unsigned*)sc)[lane] = (unsigned)(lane * 8) | ((unsigned)(lane * 8 + 4) << 16);
  BARRIER();
  asm volatile("s_waitcnt vmcnt(0)" : "+v"(d), "+v"(call_raw) :: "memory");
  const unsigned call = __builtin_amdgcn_readfirstlane(call_raw) + 1u;
  const size_t slot = WS_X + (size_t)(call & 1u) * SLOT_B;
  const unsigned xcc = (unsigned)__builtin_amdgcn_s_getreg((3 << 11) | 20) & 0xFu;
  unsigned* const tmo = (unsigned*)(ws + WS_TMO);
  unsigned lead_old = call;
  if (wid == 5 && lane == 0)
    asm volatile("global_atomic_umax %0, %1, %2, off sc0" : "+v"(lead_old) : "v"(ws + WS_LEAD + xcc * 128), "v"(call) : "memory");
  {
    const unsigned esum = (unsigned)__builtin_amdgcn_readlane(wave_scan(
        __builtin_amdgcn_udot4((d[0] >> 2) & 0x1F1F1F1Fu, 0x01010101u,
        __builtin_amdgcn_udot4((d[1] >> 2) & 0x1F1F1F1Fu, 0x01010101u,
        __builtin_amdgcn_udot4((d[2] >> 2) & 0x1F1F1F1Fu, 0x01010101u,
        __builtin_amdgcn_udot4((d[3] >> 2) & 0x1F1F1F1Fu, 0x01010101u, 0u, false), false), false), false)), 63);
    const unsigned base = esum < 4280u ? 0u : ((esum - 4280u) >> 10 > 24u ? 24u : (esum - 4280u) >> 10);
    const unsigned b4 = base * 0x04040404u;
    unsigned c[4], P[4];
#pragma unroll
    for (int i = 0; i < 4; i++) {
      const unsigned t = (d[i] | 0x80808080u) - b4;
      c[i] = (d[i] & 0x80808080u) | (t & 0x1F1F1F1Fu);
      P[i] = (c[i] + b4) ^ d[i];
    }
    const u32x3 o = {c[0] | ((c[3] & 0x03030303u) << 5), c[1] | ((c[3] & 0x0C0C0C0Cu) << 3),
                     c[2] | ((c[3] & 0x10101010u) << 1) | ((c[3] & 0x80808080u) >> 1)};
    uint8_t* trec = ws + slot + (size_t)(G >> 3) * TREC;
    st12(trec + (G & 7) * GRP_P + half * 768 + lane * 12, o);
    const unsigned z0 = (P[0] + 0x7C7C7C7Cu) & 0x80808080u, z1 = (P[1] + 0x7C7C7C7Cu) & 0x80808080u;
    const unsigned z2 = (P[2] + 0x7C7C7C7Cu) & 0x80808080u, z3 = (P[3] + 0x7C7C7C7Cu) & 0x80808080u;
    unsigned mask = (__builtin_amdgcn_udot4(z0, 0x08040201u, __builtin_amdgcn_udot4(z1, 0x80402010u, 0u, false), false) >> 7) |
                    (__builtin_amdgcn_udot4(z2, 0x08040201u, __builtin_amdgcn_udot4(z3, 0x80402010u, 0u, false), false) << 1);
    const unsigned ncnt = (unsigned)__builtin_popcount(mask);
    const unsigned incl = wave_scan(ncnt);
    const unsigned ne = (unsigned)__builtin_amdgcn_readlane(incl, 63);
    const int dir = half ? -2 : 2;
    LDS_AS uint8_t* ea = sc + (half ? 110 : 8) + (int)(incl - ncnt) * dir;
    const unsigned long long p01 = ((unsigned long long)P[1] << 32) | P[0], p23 = ((unsigned long long)P[3] << 32) | P[2];
    const unsigned pos0 = ((unsigned)half << 10) | ((unsigned)lane << 4);
    while (mask) {
      const unsigned j = (unsigned)__builtin_ctz(mask);
      const unsigned patch = (unsigned)((j & 8u ? p23 : p01) >> ((j & 7u) * 8u + 2u)) & 31u;
      *(LDS_AS unsigned short*)ea = (unsigned short)(pos0 | j | (patch << 11));
      ea += dir;
      mask &= mask - 1u;
    }
    if (lane == 0) *(LDS_AS unsigned*)(sc + 2304 + half * 4) = ne | (base << 16);
    if (wid == 0) STAMP(101);
    BARRIER();
    const unsigned m0 = *(LDS_AS unsigned*)(sc + 2304), m1 = *(LDS_AS unsigned*)(sc + 2308);
    const bool raw = (m0 & 0xFFFFu) + (m1 & 0xFFFFu) > (unsigned)ECAP;
    if (raw) st16(ws + slot + RAW_OFF + (size_t)(G * 2 + half) * 1024 + lane * 16, d);
    if (half == 0 && lane < 28) {
      unsigned v = ((LDS_AS unsigned*)sc)[lane];
      if (lane == 0) v = (m0 >> 16) * 0x04040404u | (raw ? RAW_FLAG : 0u);
      if (lane == 1) v = (m1 >> 16) * 0x04040404u;
      st4(trec + 8 * GRP_P + (G & 7) * GRP_E + lane * 4, v);
    }
    asm volatile("s_waitcnt vmcnt(0)" : "+v"(lead_old) :: "memory");
    BARRIER();
    if (wid == 0) {
      STAMP(1);
      if (lane < NR) st4(myflag, call);
      if (lane == 0) __hip_atomic_store(cnt, call, __ATOMIC_RELAXED, __HIP_MEMORY_SCOPE_AGENT);
    }
    if (wid == 5 && __builtin_amdgcn_readfirstlane(lead_old) < call) {
      __builtin_amdgcn_fence(__ATOMIC_ACQUIRE, "");
      WAITV(0);
      if (lane == 0) __hip_atomic_store((unsigned*)(ws + WS_INV + xcc * 128), call, __ATOMIC_RELAXED, __HIP_MEMORY_SCOPE_AGENT);
      STAMP(102);
    }
  }

  const int pj = wid & 3;
  unsigned wv[8];
#pragma unroll
  for (int i = 0; i < 8; i++)
    wv[i] = (unsigned)(lane >> 4) * N_ALL + (unsigned)(((lane & 15) ^ ((lane >> 4) | ((i & 1) << 2) | (((i >> 2) & 1) << 3))) * 16) +
            (unsigned)(i * 4 * N_ALL - (i & 3) * 1024);
  const u32x4 wsrd = make_srd(p.w + (size_t)pj * 32 * N_ALL + col0, (unsigned)((size_t)KD * N_ALL - (size_t)pj * 32 * N_ALL - col0));
  const unsigned wm0 = (unsigned)(LDS_W + pj * 8192);
#define W_FIRST(i, so, slotoff) asm volatile("s_add_u32 m0, %3, %4\n\ts_nop 0\n\tbuffer_load_dwordx4 %0, %1, %2 offen lds" \
      :: "v"(wv[i]), "s"(wsrd), "s"(so), "s"(wm0), "n"(slotoff) : "memory", "scc")
#define W_NEXT(i, so) asm volatile("buffer_load_dwordx4 %0, %1, %2 offen offset:%3 lds" :: "v"(wv[i]), "s"(wsrd), "s"(so), "n"(((i) & 3) * 1024) : "memory")
  if (wid >= 4) {
    const unsigned so = KST(0) * (128u * N_ALL), so1 = KST(1) * (128u * N_ALL);
    W_FIRST(0, so, 0); W_NEXT(1, so); W_NEXT(2, so); W_NEXT(3, so); W_FIRST(4, so, 4096); W_NEXT(5, so); W_NEXT(6, so); W_NEXT(7, so);
    W_FIRST(0, so1, WT_B); W_NEXT(1, so1); W_NEXT(2, so1); W_NEXT(3, so1); W_FIRST(4, so1, WT_B + 4096); W_NEXT(5, so1); W_NEXT(6, so1); W_NEXT(7, so1);
  }

  volatile LDS_AS unsigned* ctl = (volatile LDS_AS unsigned*)(smem + LDS_CTL);
  const uint8_t* flagp = ws + WS_FLAG + ((size_t)src * NWG + rb * 64 + lane) * 64;
  if (wid == 4) {
    STAMP(105);
    const bool ok = wait_words((((unsigned)lane - 2u * krot) & 63u) < 2u * (XD + 2) ? flagp : ws + WS_INV + xcc * 128, call, tmo, lane);
    STAMP(2);
    if (lane == 0) ctl[0] = ok ? 0u : 1u;
  }
  BARRIER();
  const bool bad = __builtin_amdgcn_readfirstlane(ctl[0]) != 0u;

  const int wn = wid & 3;
  float* const outp = p.out + (size_t)(src * M_SH + rb * 128 + (lane & 15)) * N_LOC + cb * 256 + wn * 64 + (lane >> 4) * 4;
  auto store_nan = [&]() {
    const float n = __builtin_nanf("");
    const v4f nv = {n, n, n, n};
    for (int mf = 0; mf < 8; mf++)
      for (int nf = 0; nf < 4; nf++) store_out16(outp + (size_t)(mf * 16) * N_LOC + nf * 16, nv);
  };
  if (bad) {
    WAITV(0);
    if (wid < 4) store_nan();
    return;
  }

  if (wid >= 4) {
    const uint8_t* const xsrc = p.ws[src] + slot;
    u32x4 xsrd = make_srd(xsrc + (size_t)rb * NKS * TREC + pj * 2 * GRP_P, (unsigned)(4 * NKS * TREC - rb * NKS * TREC - pj * 2 * GRP_P));
    const unsigned xv = (unsigned)lane * 12u;
    const int el = lane & 31;
    const unsigned ev = (unsigned)(8 * GRP_P + (2 * pj + (lane >> 5)) * GRP_E + (el < 28 ? el : 27) * 4 - pj * 2 * GRP_P);
    const unsigned emask = (el < 2 || el >= 28) ? 0u : 0xFFFFFFFFu;
    const unsigned edef = (el < 2 || el >= 28) ? (unsigned)(el * 8) | ((unsigned)(el * 8 + 4) << 16) : 0u;
    unsigned xw = (unsigned)(LDS_X + pj * 4096 + lane * 16);
    unsigned xg = (unsigned)(LDS_X + (2 * pj + (lane >> 5)) * 2048);
    asm volatile("" : "+v"(xw), "+v"(xg));
    XRegs X[4];
#define X_LD(r, j, so) asm volatile("buffer_load_dwordx3 %0, %1, %2, %3 offen offset:%4" : "=v"((r).p[j]) : "v"(xv), "s"(xsrd), "s"(so), "n"((j) * 768) : "memory")
#define X_LE(r, so) asm volatile("buffer_load_dword %0, %1, %2, %3 offen" : "=v"((r).e) : "v"(ev), "s"(xsrd), "s"(so) : "memory")
#define X_ALL(r, u) do { const unsigned so_ = KST(u) * (unsigned)TREC; X_LD(r, 0, so_); X_LD(r, 1, so_); X_LD(r, 2, so_); X_LD(r, 3, so_); X_LE(r, so_); } while (0)
    auto raw_fix = [&](int u, unsigned hA0, unsigned hB0, LDS_AS uint8_t* dst) {
      if (xsrd[2] == 0u) return;
      const uint8_t* g = xsrc + RAW_OFF + ((size_t)(rb * NKS + KST(u)) * 16 + pj * 4) * 1024;
#pragma unroll
      for (int k = 0; k < 2; k++) {
        if ((k ? hB0 : hA0) & RAW_FLAG) {
          u32x4 a, c;
          asm volatile("global_load_dwordx4 %0, %2, %3\n\t"
                       "global_load_dwordx4 %1, %2, %3 offset:1024\n\t"
                       "s_waitcnt vmcnt(0)"
                       : "=&v"(a), "=&v"(c) : "v"((unsigned)lane * 16u), "s"(g + k * 2048) : "memory");
          *(LDS_AS u32x4*)(dst + k * 2048) = a;
          *(LDS_AS u32x4*)(dst + k * 2048 + 1024) = c;
        }
      }
    };
    X_ALL(X[0], 0); X_ALL(X[1], 1); if (XD == 3) X_ALL(X[2], 2);
#define PT(x) do {} while (0)
#define PACC() do {} while (0)
#define UNP_LO(o, b) (u32x3){((o)[0] & 0x9F9F9F9Fu) + (b), ((o)[1] & 0x9F9F9F9Fu) + (b), ((o)[2] & 0x9F9F9F9Fu) + (b)}
#define UNP_HI(o, b) ((((o)[0] >> 5) & 0x03030303u) | (((o)[1] >> 3) & 0x0C0C0C0Cu) | (((o)[2] >> 1) & 0x10101010u) | (((o)[2] << 1) & 0x80808080u)) + (b)
#define PACK(j, b) do { \
      const unsigned h_ = UNP_HI(r.p[j], b); SCHED(); \
      const u32x3 l_ = UNP_LO(r.p[j], b); \
      *(LDS_AS u32x4*)(dst + (j) * 1024) = (u32x4){l_[0], l_[1], l_[2], h_}; SCHED(); } while (0)
#define W_I(i, so, off) do { if ((i) == 0) W_FIRST(0, so, off); else if ((i) == 4) W_FIRST(4, so, (off) + 4096); else W_NEXT(i, so); } while (0)
#define PROD(v, k, DO_W, DO_X, NW) do { \
      XRegs& r = X[(k)]; XRegs& rn = X[((k) + XD) % 4]; \
      const unsigned soW = KST((v) + 2) * (128u * N_ALL), soX = KST((v) + XD) * (unsigned)TREC; \
      if (DO_W) { W_I(0, soW, (((k) + 2) % 4) * WT_B); W_I(1, soW, 0); W_I(2, soW, 0); W_I(3, soW, 0); W_I(4, soW, (((k) + 2) % 4) * WT_B); W_I(5, soW, 0); W_I(6, soW, 0); W_I(7, soW, 0); } \
      if (DO_X && XPOS == 0) { X_LD(rn, 0, soX); X_LD(rn, 1, soX); X_LD(rn, 2, soX); X_LD(rn, 3, soX); X_LE(rn, soX); } \
      PT(pt0); NW; PT(pt2); \
      asm volatile("" : "+v"(r.p[0]), "+v"(r.p[1]), "+v"(r.p[2]), "+v"(r.p[3]), "+v"(r.e));       \
      const unsigned hA0 = (unsigned)__builtin_amdgcn_readlane(r.e, 0), hA1 = (unsigned)__builtin_amdgcn_readlane(r.e, 1); \
      const unsigned hB0 = (unsigned)__builtin_amdgcn_readlane(r.e, 32), hB1 = (unsigned)__builtin_amdgcn_readlane(r.e, 33); \
      LDS_AS uint8_t* dst = (LDS_AS uint8_t*)(size_t)xw + ((k) % 2) * XT_B; \
      PACK(0, hA0); PACK(1, hA1); PACK(2, hB0); \
      const unsigned e = (r.e & emask) | edef; \
      PACK(3, hB1); \
      if (DO_X && XPOS == 1) { X_LD(rn, 0, soX); X_LD(rn, 1, soX); X_LD(rn, 2, soX); X_LD(rn, 3, soX); X_LE(rn, soX); } SCHED(); \
        \
      const unsigned a0 = (e & 0x7FCu) | xg, a1 = ((e >> 16) & 0x7FCu) | xg; \
      const unsigned v0_ = ((e >> 11) & 31u) << (((e << 3) & 24u) | 2u); SCHED(); \
      const unsigned v1_ = (e >> 27) << (((e >> 13) & 24u) | 2u); \
      __hip_atomic_fetch_xor((LDS_AS unsigned*)((LDS_AS uint8_t*)(size_t)a0 + ((k) % 2) * XT_B), v0_, __ATOMIC_RELAXED, __HIP_MEMORY_SCOPE_WORKGROUP); \
      __hip_atomic_fetch_xor((LDS_AS unsigned*)((LDS_AS uint8_t*)(size_t)a1 + ((k) % 2) * XT_B), v1_, __ATOMIC_RELAXED, __HIP_MEMORY_SCOPE_WORKGROUP); \
      SCHED(); \
      if ((hA0 | hB0) & RAW_FLAG) raw_fix((v), hA0, hB0, dst); \
      PT(pt3); \
      BARRIER(); PT(pt4); PACC(); } while (0)
    WAITV(5 * (XD - 1));
    constexpr int NST = XPOS == 1 ? 21 : XD == 2 ? 26 : 31;
    for (int v0 = 0; v0 < 28; v0 += 4) {
      PROD(v0 + 0, 0, true, true, WAITV(NST));
      if (v0 == 0 && __builtin_amdgcn_readfirstlane(ctl[pj * 2048 + 1]) != 0u) xsrd[2] = 0u;
      PROD(v0 + 1, 1, true, true, WAITV(NST));
      PROD(v0 + 2, 2, true, true, WAITV(NST));
      PROD(v0 + 3, 3, true, true, WAITV(NST));
    }
    PROD(28, 0, true, true, WAITV(NST));
    PROD(29, 1, true, XD == 2, WAITV(XPOS == 1 ? 21 : 26));
    PROD(30, 2, false, false, WAITV(13));
    PROD(31, 3, false, false, WAITV(0));
    return;
  }

  const float scale = p.sx[0] * p.sw[0];
  const unsigned xaddr = (unsigned)(LDS_X + lane * 16);
  unsigned waddr[4];
  {
    const int g = lane >> 4, q = (lane & 15) >> 1;
    const int f = q | ((g & 1) << 3);
#pragma unroll
    for (int nf = 0; nf < 4; nf++)
      waddr[nf] = (unsigned)(LDS_W + (16 * g + q) * 256 + (((wn * 4 + nf) ^ f) << 4) + (lane & 1) * 8);
  }
  v4f acc[4][8];
#pragma unroll
  for (int i = 0; i < 4; i++)
#pragma unroll
    for (int j = 0; j < 8; j++) acc[i][j] = (v4f){0.f, 0.f, 0.f, 0.f};
#define READ_W(wf, nf, off) do { LDS_AS uint8_t* wp_ = smem + waddr[nf] + (off); \
    const v2i t0_ = __builtin_amdgcn_ds_read_tr8_b64_v2i32((LDS_AS v2i*)(wp_)); \
    const v2i t1_ = __builtin_amdgcn_ds_read_tr8_b64_v2i32((LDS_AS v2i*)(wp_ + 2048)); \
    const v2i t2_ = __builtin_amdgcn_ds_read_tr8_b64_v2i32((LDS_AS v2i*)(wp_ + 16384)); \
    const v2i t3_ = __builtin_amdgcn_ds_read_tr8_b64_v2i32((LDS_AS v2i*)(wp_ + 18432)); \
    wf[nf] = (v8i){t0_[0], t0_[1], t1_[0], t1_[1], t2_[0], t2_[1], t3_[0], t3_[1]}; } while (0)
#define READ_X(r, mf, off) do { LDS_AS uint8_t* xp_ = smem + xaddr + (off) + (mf) * 2048; \
    const v4i a_ = *(LDS_AS v4i*)(xp_); const v4i c_ = *(LDS_AS v4i*)(xp_ + 1024); \
    xf[r] = (v8i){a_[0], a_[1], a_[2], a_[3], c_[0], c_[1], c_[2], c_[3]}; } while (0)
#define CBARRIER(n) asm volatile("s_waitcnt lgkmcnt(%0)\n\ts_barrier" :: "n"(n) : "memory")
  if (wid == 0) {
    const bool ok = wait_words(flagp, call, tmo, lane);
    if (lane < 4) ctl[lane * 2048 + 1] = ok ? 0u : 1u;
    asm volatile("s_waitcnt lgkmcnt(0)" ::: "memory");
  }
  unsigned tmo_late;
  v8i wf[4], xf[8];
#define MM(nf, mf) acc[nf][mf] = __builtin_amdgcn_mfma_scale_f32_16x16x128_f8f6f4(wf[nf], xf[mf], acc[nf][mf], 1, 1, 0, 0, 0, 0)
#define MROW_NR(mf) do { MM(0, mf); MM(1, mf); MM(2, mf); MM(3, mf); } while (0)
#define MROW(mf, xo) do { MROW_NR(mf); SCHED(); READ_X(mf, mf, xo); SCHED(); } while (0)
#define STEP_NB(xo, wo) do { \
    MROW(0, xo); MROW(1, xo); MROW(2, xo); MROW(3, xo); MROW(4, xo); MROW(5, xo); MROW(6, xo); \
    MM(0, 7); SCHED(); READ_W(wf, 0, wo); SCHED(); MM(1, 7); SCHED(); READ_W(wf, 1, wo); SCHED(); \
    MM(2, 7); SCHED(); READ_W(wf, 2, wo); SCHED(); MM(3, 7); SCHED(); READ_W(wf, 3, wo); READ_X(7, 7, xo); SCHED(); } while (0)
  RAW_BARRIER();
  if (wid == 0) STAMP(4);
#pragma unroll
  for (int i = 0; i < 4; i++) READ_W(wf, i, 0);
#pragma unroll
  for (int i = 0; i < 8; i++) READ_X(i, i, 0);
  BARRIER();
  asm volatile("global_load_dword %0, %1, off sc1" : "=v"(tmo_late) : "v"(tmo) : "memory");
  for (int u = 0; u < NKS - 3; u++) {
    STEP_NB(((u + 1) & 1) * XT_B, ((u + 1) & 3) * WT_B); CBARRIER(0);
    if (wid == 0 && (u % 6) == 0) STAMP(8 + u);
  }
  asm volatile("s_waitcnt vmcnt(0)" : "+v"(tmo_late) :: "memory");
  if (__builtin_amdgcn_readfirstlane(tmo_late) == call) { store_nan(); return; }
  {
    STEP_NB(((NKS - 2) % 2) * XT_B, ((NKS - 2) % 4) * WT_B);
    constexpr int xo = ((NKS - 1) % 2) * XT_B, wo = ((NKS - 1) % 4) * WT_B;
    SCHED();
    MROW_NR(0); MROW_NR(1); MROW_NR(2); MROW_NR(3); MROW_NR(4); MROW_NR(5); MROW_NR(6); MROW_NR(7);
    SCHED();
    RAW_BARRIER();
    READ_W(wf, 0, wo); READ_X(0, 0, xo); READ_W(wf, 1, wo); READ_W(wf, 2, wo); READ_W(wf, 3, wo); READ_X(1, 1, xo); SCHED();
#define EPILOGUE(mf) do { _Pragma("unroll") for (int nf = 0; nf < 4; nf++) { \
      v4f y = acc[nf][mf] * scale; \
      y[0] = fmaxf(y[0], 0.f); y[1] = fmaxf(y[1], 0.f); y[2] = fmaxf(y[2], 0.f); y[3] = fmaxf(y[3], 0.f); \
      store_out16(outp + (size_t)((mf) * 16) * N_LOC + nf * 16, y); } SCHED(); } while (0)
    MROW_NR(0); SCHED(); READ_X(2, 2, xo); SCHED(); MROW_NR(1); SCHED(); READ_X(3, 3, xo); SCHED(); EPILOGUE(0);
    MROW_NR(2); SCHED(); READ_X(4, 4, xo); SCHED(); EPILOGUE(1); MROW_NR(3); SCHED(); READ_X(5, 5, xo); SCHED(); EPILOGUE(2);
    MROW_NR(4); SCHED(); READ_X(6, 6, xo); SCHED(); EPILOGUE(3); MROW_NR(5); SCHED(); READ_X(7, 7, xo); SCHED(); EPILOGUE(4);
    MROW_NR(6); SCHED(); EPILOGUE(5); MROW_NR(7); SCHED(); EPILOGUE(6); EPILOGUE(7);
  }
}

}

extern "C" void kernel_launch(int rank, int nranks, ncclComm_t comm,
                              hipStream_t stream,
                              void* const* d_in, const int* in_sizes, int n_in,
                              void* d_out, int out_size,
                              const int* in_dtypes, int out_dtype,
                              void* const* d_out_all,
                              void* const* d_ws_all, size_t ws_bytes) {
  (void)comm; (void)in_sizes; (void)in_dtypes; (void)out_dtype; (void)d_out_all; (void)out_size;
  if (nranks != NR || n_in != 4 || ws_bytes < WS_NEED || (unsigned)rank >= (unsigned)NR) {
    fprintf(stderr, "kernel_launch: unexpected configuration\n");
    return;
  }
  static bool init = false;
  if (!init) {
    HIP_CHECK(hipFuncSetAttribute((const void*)gemm_ag_kernel, hipFuncAttributeMaxDynamicSharedMemorySize, LDS_BYTES));
    init = true;
  }
  Params p;
  p.x = (const uint8_t*)d_in[0]; p.w = (const uint8_t*)d_in[1];
  p.sx = (const float*)d_in[2]; p.sw = (const float*)d_in[3];
  p.out = (float*)d_out;
  for (int i = 0; i < NR; i++) p.ws[i] = (uint8_t*)d_ws_all[i];
  p.rank = rank; p.pad = 0;
  gemm_ag_kernel<<<NWG, NTHR, LDS_BYTES, stream>>>(p);
  HIP_CHECK(hipGetLastError());
}
```

```cpp
#include <hip/hip_runtime.h>
#include <rccl/rccl.h>
#include <cstdio>
#include <cstdint>

#define HIP_CHECK(expr) do { hipError_t _e = (expr); if (_e != hipSuccess) { \
    fprintf(stderr, "HIP error %s at %s:%d\n", hipGetErrorString(_e), __FILE__, __LINE__); \
    return; } } while (0)

namespace {

typedef int      v8i   __attribute__((ext_vector_type(8)));
typedef int      v4i   __attribute__((ext_vector_type(4)));
typedef int      v2i   __attribute__((ext_vector_type(2)));
typedef float    v4f   __attribute__((ext_vector_type(4)));
typedef unsigned u32x4 __attribute__((ext_vector_type(4)));
typedef unsigned u32x3 __attribute__((ext_vector_type(3)));
#define LDS_AS __attribute__((address_space(3)))

constexpr int NR = 8;
constexpr int KD = 4096;
constexpr int N_ALL = 16384;
constexpr int N_LOC = 2048;
constexpr int M_SH = 512;
constexpr int NWG = 256;
constexpr int NTHR = 512;
constexpr int NKS = 32;
constexpr int XT_B = 16384;
constexpr int WT_B = 32768;
constexpr int LDS_X = 0;
constexpr int LDS_W = 2 * XT_B;
constexpr int LDS_BYTES = LDS_W + 4 * WT_B;
constexpr int LDS_CTL = LDS_W + 3 * WT_B;

constexpr int GRP_P = 1536, GRP_E = 112, ECAP = 52;
constexpr int TREC = 8 * (GRP_P + GRP_E);
constexpr unsigned RAW_FLAG = 0x80u;
constexpr int SCR_B = 4608;

constexpr size_t WS_CNT  = 0;
constexpr size_t WS_TMO  = WS_CNT + NWG * 128;
constexpr size_t WS_LEAD = WS_TMO + 128;
constexpr size_t WS_INV  = WS_LEAD + 16 * 128;
constexpr size_t WS_FLAG = WS_INV + 16 * 128 + 3968;
constexpr size_t WS_X    = 262144;
constexpr size_t RAW_OFF = 1703936;
constexpr size_t SLOT_B  = RAW_OFF + (size_t)M_SH * KD;
constexpr size_t WS_NEED = WS_X + 2 * SLOT_B;
static_assert(WS_FLAG % 128 == 0 && WS_FLAG + NR * NWG * 64 <= WS_X, "ws layout");
static_assert(4 * NKS * TREC <= RAW_OFF, "ws layout");

constexpr unsigned long long DEADLINE_TICKS = 30000000ull;

struct Params {
  const uint8_t* x; const uint8_t* w; const float* sx; const float* sw; float* out;
  uint8_t* ws[NR];
  int rank; int pad;
};

#define WAITV(n) asm volatile("s_waitcnt vmcnt(%0)" :: "n"(n) : "memory")
#define BARRIER() asm volatile("s_waitcnt lgkmcnt(0)\n\ts_barrier" ::: "memory")
#define RAW_BARRIER() asm volatile("s_barrier" ::: "memory")
#define SCHED() __builtin_amdgcn_sched_barrier(0)

__device__ __forceinline__ void st16(void* p, u32x4 v) { asm volatile("global_store_dwordx4 %0, %1, off sc0 sc1\n\ts_nop 1" :: "v"(p), "v"(v) : "memory"); }
__device__ __forceinline__ void st12(void* p, u32x3 v) { asm volatile("global_store_dwordx3 %0, %1, off sc0 sc1\n\ts_nop 1" :: "v"(p), "v"(v) : "memory"); }
__device__ __forceinline__ void st4(void* p, unsigned v) { asm volatile("global_store_dword %0, %1, off sc0 sc1" :: "v"(p), "v"(v) : "memory"); }
__device__ __forceinline__ void store_out16(void* p, v4f v) { asm volatile("global_store_dwordx4 %0, %1, off sc0 sc1\n\ts_nop 1" :: "v"(p), "v"(v) : "memory"); }

__device__ __forceinline__ unsigned wave_scan(unsigned v) {
  v += __builtin_amdgcn_update_dpp(0u, v, 0x111, 0xf, 0xf, true);
  v += __builtin_amdgcn_update_dpp(0u, v, 0x112, 0xf, 0xf, true);
  v += __builtin_amdgcn_update_dpp(0u, v, 0x114, 0xf, 0xf, true);
  v += __builtin_amdgcn_update_dpp(0u, v, 0x118, 0xf, 0xf, true);
  v += __builtin_amdgcn_update_dpp(0u, v, 0x142, 0xa, 0xf, true);
  v += __builtin_amdgcn_update_dpp(0u, v, 0x143, 0xc, 0xf, true);
  return v;
}

__device__ __forceinline__ u32x4 make_srd(const void* base, unsigned bytes) {
  const unsigned long long a = (unsigned long long)base;
  return (u32x4){(unsigned)__builtin_amdgcn_readfirstlane((unsigned)a), (unsigned)__builtin_amdgcn_readfirstlane((unsigned)(a >> 32)) & 0xFFFFu,
                 (unsigned)__builtin_amdgcn_readfirstlane(bytes), 0x00020000u};
}

__device__ __forceinline__ bool wait_words(const uint8_t* addr, unsigned call, unsigned* tmo, int lane) {
  const unsigned long long t0 = __builtin_amdgcn_s_memrealtime();
  while (true) {
    unsigned v, t;
    asm volatile("global_load_dword %0, %2, off sc0 sc1\n\tglobal_load_dword %1, %3, off sc1\n\ts_waitcnt vmcnt(0)"
                 : "=&v"(v), "=&v"(t) : "v"(addr), "v"(tmo) : "memory");
    if (__all(v >= call)) return true;
    if ((unsigned)__builtin_amdgcn_readfirstlane(t) == call || __builtin_amdgcn_s_memrealtime() - t0 > DEADLINE_TICKS) {
      if (lane == 0) __hip_atomic_store(tmo, call, __ATOMIC_RELAXED, __HIP_MEMORY_SCOPE_AGENT);
      return false;
    }
    __builtin_amdgcn_s_sleep(2);
  }
}

struct XRegs { u32x3 p[4]; unsigned e; };

__global__ __launch_bounds__(NTHR, 2) void gemm_ag_kernel(Params p) {
  extern __shared__ __attribute__((aligned(16))) uint8_t smem_[];
  LDS_AS uint8_t* smem = (LDS_AS uint8_t*)smem_;
  const int tid = threadIdx.x, lane = tid & 63;
  const int wid = __builtin_amdgcn_readfirstlane(tid >> 6);
  const int b = blockIdx.x;
  uint8_t* const ws = p.ws[p.rank];

  unsigned* cnt = (unsigned*)(ws + WS_CNT + (size_t)b * 128);
  const int G = b * 4 + (wid >> 1), half = wid & 1;
  const uint8_t* gx = p.x + (size_t)((G >> 8) * 128 + (G & 7) * 16 + (lane & 15)) * KD + ((G >> 3) & 31) * 128 + half * 64 + (lane >> 4) * 16;
  unsigned call_raw; u32x4 d;
  asm volatile("global_load_dwordx4 %0, %2, off\n\tglobal_load_dword %1, %3, off sc1" : "=&v"(d), "=&v"(call_raw) : "v"(gx), "v"(cnt) : "memory");

  const int grp = b & 7, idx = b >> 3;
  const int src = (p.rank + grp) & (NR - 1);
  const int rb = idx >> 3, cb = idx & 7;
  const int col0 = p.rank * N_LOC + cb * 256;
  const unsigned krot = (unsigned)(grp * 4);
#define KST(u) (((unsigned)(u) + krot) & 31u)

  unsigned long long peer = (unsigned long long)p.ws[0];
#pragma unroll
  for (int i = 1; i < NR; i++) {
    unsigned long long wi = (unsigned long long)p.ws[i];
    asm volatile("" : "+s"(wi));
    peer = (lane == i) ? wi : peer;
  }
  uint8_t* const myflag = (uint8_t*)peer + WS_FLAG + ((size_t)p.rank * NWG + b) * 64;

  LDS_AS uint8_t* sc = smem + LDS_X + (wid >> 1) * SCR_B + 2048;
  if (half == 0 && lane < 28) ((LDS_AS unsigned*)sc)[lane] = (unsigned)(lane * 8) | ((unsigned)(lane * 8 + 4) << 16);
  BARRIER();
  asm volatile("s_waitcnt vmcnt(0)" : "+v"(d), "+v"(call_raw) :: "memory");
  const unsigned call = __builtin_amdgcn_readfirstlane(call_raw) + 1u;
  const size_t slot = WS_X + (size_t)(call & 1u) * SLOT_B;
  const unsigned xcc = (unsigned)__builtin_amdgcn_s_getreg((3 << 11) | 20) & 0xFu;
  unsigned* const tmo = (unsigned*)(ws + WS_TMO);
  unsigned lead_old = call;
  if (wid == 5 && lane == 0)
    asm volatile("global_atomic_umax %0, %1, %2, off sc0" : "+v"(lead_old) : "v"(ws + WS_LEAD + xcc * 128), "v"(call) : "memory");
  {
    const unsigned esum = (unsigned)__builtin_amdgcn_readlane(wave_scan(
        __builtin_amdgcn_udot4((d[0] >> 2) & 0x1F1F1F1Fu, 0x01010101u,
        __builtin_amdgcn_udot4((d[1] >> 2) & 0x1F1F1F1Fu, 0x01010101u,
        __builtin_amdgcn_udot4((d[2] >> 2) & 0x1F1F1F1Fu, 0x01010101u,
        __builtin_amdgcn_udot4((d[3] >> 2) & 0x1F1F1F1Fu, 0x01010101u, 0u, false), false), false), false)), 63);
    const unsigned base = esum < 4280u ? 0u : ((esum - 4280u) >> 10 > 24u ? 24u : (esum - 4280u) >> 10);
    const unsigned b4 = base * 0x04040404u;
    unsigned c[4], P[4];
#pragma unroll
    for (int i = 0; i < 4; i++) {
      const unsigned t = (d[i] | 0x80808080u) - b4;
      c[i] = (d[i] & 0x80808080u) | (t & 0x1F1F1F1Fu);
      P[i] = (c[i] + b4) ^ d[i];
    }
    const u32x3 o = {c[0] | ((c[3] & 0x03030303u) << 5), c[1] | ((c[3] & 0x0C0C0C0Cu) << 3),
                     c[2] | ((c[3] & 0x10101010u) << 1) | ((c[3] & 0x80808080u) >> 1)};
    uint8_t* trec = ws + slot + (size_t)(G >> 3) * TREC;
    st12(trec + (G & 7) * GRP_P + half * 768 + lane * 12, o);
    const unsigned z0 = (P[0] + 0x7C7C7C7Cu) & 0x80808080u, z1 = (P[1] + 0x7C7C7C7Cu) & 0x80808080u;
    const unsigned z2 = (P[2] + 0x7C7C7C7Cu) & 0x80808080u, z3 = (P[3] + 0x7C7C7C7Cu) & 0x80808080u;
    unsigned mask = (__builtin_amdgcn_udot4(z0, 0x08040201u, __builtin_amdgcn_udot4(z1, 0x80402010u, 0u, false), false) >> 7) |
                    (__builtin_amdgcn_udot4(z2, 0x08040201u, __builtin_amdgcn_udot4(z3, 0x80402010u, 0u, false), false) << 1);
    const unsigned ncnt = (unsigned)__builtin_popcount(mask);
    const unsigned incl = wave_scan(ncnt);
    const unsigned ne = (unsigned)__builtin_amdgcn_readlane(incl, 63);
    const int dir = half ? -2 : 2;
    LDS_AS uint8_t* ea = sc + (half ? 110 : 8) + (int)(incl - ncnt) * dir;
    const unsigned long long p01 = ((unsigned long long)P[1] << 32) | P[0], p23 = ((unsigned long long)P[3] << 32) | P[2];
    const unsigned pos0 = ((unsigned)half << 10) | ((unsigned)lane << 4);
    while (mask) {
      const unsigned j = (unsigned)__builtin_ctz(mask);
      const unsigned patch = (unsigned)((j & 8u ? p23 : p01) >> ((j & 7u) * 8u + 2u)) & 31u;
      *(LDS_AS unsigned short*)ea = (unsigned short)(pos0 | j | (patch << 11));
      ea += dir;
      mask &= mask - 1u;
    }
    if (lane == 0) *(LDS_AS unsigned*)(sc + 2304 + half * 4) = ne | (base << 16);
    BARRIER();
    const unsigned m0 = *(LDS_AS unsigned*)(sc + 2304), m1 = *(LDS_AS unsigned*)(sc + 2308);
    const bool raw = (m0 & 0xFFFFu) + (m1 & 0xFFFFu) > (unsigned)ECAP;
    if (raw) st16(ws + slot + RAW_OFF + (size_t)(G * 2 + half) * 1024 + lane * 16, d);
    if (half == 0 && lane < 28) {
      unsigned v = ((LDS_AS unsigned*)sc)[lane];
      if (lane == 0) v = (m0 >> 16) * 0x04040404u | (raw ? RAW_FLAG : 0u);
      if (lane == 1) v = (m1 >> 16) * 0x04040404u;
      st4(trec + 8 * GRP_P + (G & 7) * GRP_E + lane * 4, v);
    }
    asm volatile("s_waitcnt vmcnt(0)" : "+v"(lead_old) :: "memory");
    BARRIER();
    if (wid == 0) {
      if (lane < NR) st4(myflag, call);
      if (lane == 0) __hip_atomic_store(cnt, call, __ATOMIC_RELAXED, __HIP_MEMORY_SCOPE_AGENT);
    }
    if (wid == 5 && __builtin_amdgcn_readfirstlane(lead_old) < call) {
      __builtin_amdgcn_fence(__ATOMIC_ACQUIRE, "");
      WAITV(0);
      if (lane == 0) __hip_atomic_store((unsigned*)(ws + WS_INV + xcc * 128), call, __ATOMIC_RELAXED, __HIP_MEMORY_SCOPE_AGENT);
    }
  }

  const int pj = wid & 3;
  unsigned wv[8];
#pragma unroll
  for (int i = 0; i < 8; i++)
    wv[i] = (unsigned)(lane >> 4) * N_ALL + (unsigned)(((lane & 15) ^ ((lane >> 4) | ((i & 1) << 2) | (((i >> 2) & 1) << 3))) * 16) +
            (unsigned)(i * 4 * N_ALL - (i & 3) * 1024);
  const u32x4 wsrd = make_srd(p.w + (size_t)pj * 32 * N_ALL + col0, (unsigned)((size_t)KD * N_ALL - (size_t)pj * 32 * N_ALL - col0));
  const unsigned wm0 = (unsigned)(LDS_W + pj * 8192);
#define W_FIRST(i, so, slotoff) asm volatile("s_add_u32 m0, %3, %4\n\ts_nop 0\n\tbuffer_load_dwordx4 %0, %1, %2 offen lds" \
      :: "v"(wv[i]), "s"(wsrd), "s"(so), "s"(wm0), "n"(slotoff) : "memory", "scc")
#define W_NEXT(i, so) asm volatile("buffer_load_dwordx4 %0, %1, %2 offen offset:%3 lds" :: "v"(wv[i]), "s"(wsrd), "s"(so), "n"(((i) & 3) * 1024) : "memory")
  if (wid >= 4) {
    const unsigned so = KST(0) * (128u * N_ALL), so1 = KST(1) * (128u * N_ALL);
    W_FIRST(0, so, 0); W_NEXT(1, so); W_NEXT(2, so); W_NEXT(3, so); W_FIRST(4, so, 4096); W_NEXT(5, so); W_NEXT(6, so); W_NEXT(7, so);
    W_FIRST(0, so1, WT_B); W_NEXT(1, so1); W_NEXT(2, so1); W_NEXT(3, so1); W_FIRST(4, so1, WT_B + 4096); W_NEXT(5, so1); W_NEXT(6, so1); W_NEXT(7, so1);
  }

  volatile LDS_AS unsigned* ctl = (volatile LDS_AS unsigned*)(smem + LDS_CTL);
  const uint8_t* flagp = ws + WS_FLAG + ((size_t)src * NWG + rb * 64 + lane) * 64;
  if (wid == 4) {
    const bool ok = wait_words((((unsigned)lane - 2u * krot) & 63u) < 8u ? flagp : ws + WS_INV + xcc * 128, call, tmo, lane);
    if (lane == 0) ctl[0] = ok ? 0u : 1u;
  }
  BARRIER();
  const bool bad = __builtin_amdgcn_readfirstlane(ctl[0]) != 0u;

  const int wn = wid & 3;
  float* const outp = p.out + (size_t)(src * M_SH + rb * 128 + (lane & 15)) * N_LOC + cb * 256 + wn * 64 + (lane >> 4) * 4;
  auto store_nan = [&]() {
    const float n = __builtin_nanf("");
    const v4f nv = {n, n, n, n};
    for (int mf = 0; mf < 8; mf++)
      for (int nf = 0; nf < 4; nf++) store_out16(outp + (size_t)(mf * 16) * N_LOC + nf * 16, nv);
  };
  if (bad) {
    WAITV(0);
    if (wid < 4) store_nan();
    return;
  }

  if (wid >= 4) {
    const uint8_t* const xsrc = p.ws[src] + slot;
    u32x4 xsrd = make_srd(xsrc + (size_t)rb * NKS * TREC + pj * 2 * GRP_P, (unsigned)(4 * NKS * TREC - rb * NKS * TREC - pj * 2 * GRP_P));
    const unsigned xv = (unsigned)lane * 12u;
    const int el = lane & 31;
    const unsigned ev = (unsigned)(8 * GRP_P + (2 * pj + (lane >> 5)) * GRP_E + (el < 28 ? el : 27) * 4 - pj * 2 * GRP_P);
    const unsigned emask = (el < 2 || el >= 28) ? 0u : 0xFFFFFFFFu;
    const unsigned edef = (el < 2 || el >= 28) ? (unsigned)(el * 8) | ((unsigned)(el * 8 + 4) << 16) : 0u;
    unsigned xw = (unsigned)(LDS_X + pj * 4096 + lane * 16);
    unsigned xg = (unsigned)(LDS_X + (2 * pj + (lane >> 5)) * 2048);
    asm volatile("" : "+v"(xw), "+v"(xg));
    XRegs X[4];
#define X_LD(r, j, so) asm volatile("buffer_load_dwordx3 %0, %1, %2, %3 offen offset:%4" : "=v"((r).p[j]) : "v"(xv), "s"(xsrd), "s"(so), "n"((j) * 768) : "memory")
#define X_LE(r, so) asm volatile("buffer_load_dword %0, %1, %2, %3 offen" : "=v"((r).e) : "v"(ev), "s"(xsrd), "s"(so) : "memory")
#define X_ALL(r, u) do { const unsigned so_ = KST(u) * (unsigned)TREC; X_LD(r, 0, so_); X_LD(r, 1, so_); X_LD(r, 2, so_); X_LD(r, 3, so_); X_LE(r, so_); } while (0)
    auto raw_fix = [&](int u, unsigned hA0, unsigned hB0, LDS_AS uint8_t* dst) {
      if (xsrd[2] == 0u) return;
      const uint8_t* g = xsrc + RAW_OFF + ((size_t)(rb * NKS + KST(u)) * 16 + pj * 4) * 1024;
#pragma unroll
      for (int k = 0; k < 2; k++) {
        if ((k ? hB0 : hA0) & RAW_FLAG) {
          u32x4 a, c;
          asm volatile("global_load_dwordx4 %0, %2, %3\n\t"
                       "global_load_dwordx4 %1, %2, %3 offset:1024\n\t"
                       "s_waitcnt vmcnt(0)"
                       : "=&v"(a), "=&v"(c) : "v"((unsigned)lane * 16u), "s"(g + k * 2048) : "memory");
          *(LDS_AS u32x4*)(dst + k * 2048) = a;
          *(LDS_AS u32x4*)(dst + k * 2048 + 1024) = c;
        }
      }
    };
    X_ALL(X[0], 0); X_ALL(X[1], 1);
#define UNP_LO(o, b) (u32x3){((o)[0] & 0x9F9F9F9Fu) + (b), ((o)[1] & 0x9F9F9F9Fu) + (b), ((o)[2] & 0x9F9F9F9Fu) + (b)}
#define UNP_HI(o, b) ((((o)[0] >> 5) & 0x03030303u) | (((o)[1] >> 3) & 0x0C0C0C0Cu) | (((o)[2] >> 1) & 0x10101010u) | (((o)[2] << 1) & 0x80808080u)) + (b)
#define PACK(j, b) do { \
      const unsigned h_ = UNP_HI(r.p[j], b); SCHED(); \
      const u32x3 l_ = UNP_LO(r.p[j], b); \
      *(LDS_AS u32x4*)(dst + (j) * 1024) = (u32x4){l_[0], l_[1], l_[2], h_}; SCHED(); } while (0)
#define W_I(i, so, off) do { if ((i) == 0) W_FIRST(0, so, off); else if ((i) == 4) W_FIRST(4, so, (off) + 4096); else W_NEXT(i, so); } while (0)
#define PROD(v, k, DO_W, DO_X, NW) do { \
      XRegs& r = X[(k)]; XRegs& rn = X[((k) + 2) % 4]; \
      const unsigned soW = KST((v) + 2) * (128u * N_ALL), soX = KST((v) + 2) * (unsigned)TREC; \
      if (DO_W) { W_I(0, soW, (((k) + 2) % 4) * WT_B); W_I(1, soW, 0); W_I(2, soW, 0); W_I(3, soW, 0); W_I(4, soW, (((k) + 2) % 4) * WT_B); W_I(5, soW, 0); W_I(6, soW, 0); W_I(7, soW, 0); } \
      if (DO_X) { X_LD(rn, 0, soX); X_LD(rn, 1, soX); X_LD(rn, 2, soX); X_LD(rn, 3, soX); X_LE(rn, soX); } \
      NW; \
      asm volatile("" : "+v"(r.p[0]), "+v"(r.p[1]), "+v"(r.p[2]), "+v"(r.p[3]), "+v"(r.e));       \
      const unsigned hA0 = (unsigned)__builtin_amdgcn_readlane(r.e, 0), hA1 = (unsigned)__builtin_amdgcn_readlane(r.e, 1); \
      const unsigned hB0 = (unsigned)__builtin_amdgcn_readlane(r.e, 32), hB1 = (unsigned)__builtin_amdgcn_readlane(r.e, 33); \
      LDS_AS uint8_t* dst = (LDS_AS uint8_t*)(size_t)xw + ((k) % 2) * XT_B; \
      PACK(0, hA0); PACK(1, hA1); PACK(2, hB0); \
      const unsigned e = (r.e & emask) | edef; \
      PACK(3, hB1); \
        \
      const unsigned a0 = (e & 0x7FCu) | xg, a1 = ((e >> 16) & 0x7FCu) | xg; \
      const unsigned v0_ = ((e >> 11) & 31u) << (((e << 3) & 24u) | 2u); SCHED(); \
      const unsigned v1_ = (e >> 27) << (((e >> 13) & 24u) | 2u); \
      __hip_atomic_fetch_xor((LDS_AS unsigned*)((LDS_AS uint8_t*)(size_t)a0 + ((k) % 2) * XT_B), v0_, __ATOMIC_RELAXED, __HIP_MEMORY_SCOPE_WORKGROUP); \
      __hip_atomic_fetch_xor((LDS_AS unsigned*)((LDS_AS uint8_t*)(size_t)a1 + ((k) % 2) * XT_B), v1_, __ATOMIC_RELAXED, __HIP_MEMORY_SCOPE_WORKGROUP); \
      SCHED(); \
      if ((hA0 | hB0) & RAW_FLAG) raw_fix((v), hA0, hB0, dst); \
      BARRIER(); } while (0)
    WAITV(5);
    for (int v0 = 0; v0 < 28; v0 += 4) {
      PROD(v0 + 0, 0, true, true, WAITV(26));
      if (v0 == 0 && __builtin_amdgcn_readfirstlane(ctl[pj * 2048 + 1]) != 0u) xsrd[2] = 0u;
      PROD(v0 + 1, 1, true, true, WAITV(26));
      PROD(v0 + 2, 2, true, true, WAITV(26));
      PROD(v0 + 3, 3, true, true, WAITV(26));
    }
    PROD(28, 0, true, true, WAITV(26));
    PROD(29, 1, true, true, WAITV(26));
    PROD(30, 2, false, false, WAITV(13));
    PROD(31, 3, false, false, WAITV(0));
    return;
  }

  const float scale = p.sx[0] * p.sw[0];
  const unsigned xaddr = (unsigned)(LDS_X + lane * 16);
  unsigned waddr[4];
  {
    const int g = lane >> 4, q = (lane & 15) >> 1;
    const int f = q | ((g & 1) << 3);
#pragma unroll
    for (int nf = 0; nf < 4; nf++)
      waddr[nf] = (unsigned)(LDS_W + (16 * g + q) * 256 + (((wn * 4 + nf) ^ f) << 4) + (lane & 1) * 8);
  }
  v4f acc[4][8];
#pragma unroll
  for (int i = 0; i < 4; i++)
#pragma unroll
    for (int j = 0; j < 8; j++) acc[i][j] = (v4f){0.f, 0.f, 0.f, 0.f};
#define READ_W(wf, nf, off) do { LDS_AS uint8_t* wp_ = smem + waddr[nf] + (off); \
    const v2i t0_ = __builtin_amdgcn_ds_read_tr8_b64_v2i32((LDS_AS v2i*)(wp_)); \
    const v2i t1_ = __builtin_amdgcn_ds_read_tr8_b64_v2i32((LDS_AS v2i*)(wp_ + 2048)); \
    const v2i t2_ = __builtin_amdgcn_ds_read_tr8_b64_v2i32((LDS_AS v2i*)(wp_ + 16384)); \
    const v2i t3_ = __builtin_amdgcn_ds_read_tr8_b64_v2i32((LDS_AS v2i*)(wp_ + 18432)); \
    wf[nf] = (v8i){t0_[0], t0_[1], t1_[0], t1_[1], t2_[0], t2_[1], t3_[0], t3_[1]}; } while (0)
#define READ_X(r, mf, off) do { LDS_AS uint8_t* xp_ = smem + xaddr + (off) + (mf) * 2048; \
    const v4i a_ = *(LDS_AS v4i*)(xp_); const v4i c_ = *(LDS_AS v4i*)(xp_ + 1024); \
    xf[r] = (v8i){a_[0], a_[1], a_[2], a_[3], c_[0], c_[1], c_[2], c_[3]}; } while (0)
  if (wid == 0) {
    const bool ok = wait_words(flagp, call, tmo, lane);
    if (lane < 4) ctl[lane * 2048 + 1] = ok ? 0u : 1u;
    asm volatile("s_waitcnt lgkmcnt(0)" ::: "memory");
  }
  unsigned tmo_late;
  v8i wf[4], xf[8];
#define MM(nf, mf) acc[nf][mf] = __builtin_amdgcn_mfma_scale_f32_16x16x128_f8f6f4(wf[nf], xf[mf], acc[nf][mf], 1, 1, 0, 0, 0, 0)
#define MROW_NR(mf) do { MM(0, mf); MM(1, mf); MM(2, mf); MM(3, mf); } while (0)
#define MROW(mf, xo) do { MROW_NR(mf); SCHED(); READ_X(mf, mf, xo); SCHED(); } while (0)
#define STEP_NB(xo, wo) do { \
    MROW(0, xo); MROW(1, xo); MROW(2, xo); MROW(3, xo); MROW(4, xo); MROW(5, xo); MROW(6, xo); \
    MM(0, 7); SCHED(); READ_W(wf, 0, wo); SCHED(); MM(1, 7); SCHED(); READ_W(wf, 1, wo); SCHED(); \
    MM(2, 7); SCHED(); READ_W(wf, 2, wo); SCHED(); MM(3, 7); SCHED(); READ_W(wf, 3, wo); READ_X(7, 7, xo); SCHED(); } while (0)
  RAW_BARRIER();
#pragma unroll
  for (int i = 0; i < 4; i++) READ_W(wf, i, 0);
#pragma unroll
  for (int i = 0; i < 8; i++) READ_X(i, i, 0);
  BARRIER();
  asm volatile("global_load_dword %0, %1, off sc1" : "=v"(tmo_late) : "v"(tmo) : "memory");
  for (int u = 0; u < NKS - 3; u++) {
    STEP_NB(((u + 1) & 1) * XT_B, ((u + 1) & 3) * WT_B); BARRIER();
  }
  asm volatile("s_waitcnt vmcnt(0)" : "+v"(tmo_late) :: "memory");
  if (__builtin_amdgcn_readfirstlane(tmo_late) == call) { store_nan(); return; }
  {
    STEP_NB(((NKS - 2) % 2) * XT_B, ((NKS - 2) % 4) * WT_B);
    constexpr int xo = ((NKS - 1) % 2) * XT_B, wo = ((NKS - 1) % 4) * WT_B;
    SCHED();
    MROW_NR(0); MROW_NR(1); MROW_NR(2); MROW_NR(3); MROW_NR(4); MROW_NR(5); MROW_NR(6); MROW_NR(7);
    SCHED();
    RAW_BARRIER();
    READ_W(wf, 0, wo); READ_X(0, 0, xo); READ_W(wf, 1, wo); READ_W(wf, 2, wo); READ_W(wf, 3, wo); READ_X(1, 1, xo); SCHED();
#define EPILOGUE(mf) do { _Pragma("unroll") for (int nf = 0; nf < 4; nf++) { \
      v4f y = acc[nf][mf] * scale; \
      y[0] = fmaxf(y[0], 0.f); y[1] = fmaxf(y[1], 0.f); y[2] = fmaxf(y[2], 0.f); y[3] = fmaxf(y[3], 0.f); \
      store_out16(outp + (size_t)((mf) * 16) * N_LOC + nf * 16, y); } SCHED(); } while (0)
    MROW_NR(0); SCHED(); READ_X(2, 2, xo); SCHED(); MROW_NR(1); SCHED(); READ_X(3, 3, xo); SCHED(); EPILOGUE(0);
    MROW_NR(2); SCHED(); READ_X(4, 4, xo); SCHED(); EPILOGUE(1); MROW_NR(3); SCHED(); READ_X(5, 5, xo); SCHED(); EPILOGUE(2);
    MROW_NR(4); SCHED(); READ_X(6, 6, xo); SCHED(); EPILOGUE(3); MROW_NR(5); SCHED(); READ_X(7, 7, xo); SCHED(); EPILOGUE(4);
    MROW_NR(6); SCHED(); EPILOGUE(5); MROW_NR(7); SCHED(); EPILOGUE(6); EPILOGUE(7);
  }
}

}

extern "C" void kernel_launch(int rank, int nranks, ncclComm_t comm,
                              hipStream_t stream,
                              void* const* d_in, const int* in_sizes, int n_in,
                              void* d_out, int out_size,
                              const int* in_dtypes, int out_dtype,
                              void* const* d_out_all,
                              void* const* d_ws_all, size_t ws_bytes) {
  (void)comm; (void)in_sizes; (void)in_dtypes; (void)out_dtype; (void)d_out_all; (void)out_size;
  if (nranks != NR || n_in != 4 || ws_bytes < WS_NEED || (unsigned)rank >= (unsigned)NR) {
    fprintf(stderr, "kernel_launch: unexpected configuration\n");
    return;
  }
  static bool init = false;
  if (!init) {
    HIP_CHECK(hipFuncSetAttribute((const void*)gemm_ag_kernel, hipFuncAttributeMaxDynamicSharedMemorySize, LDS_BYTES));
    init = true;
  }
  Params p;
  p.x = (const uint8_t*)d_in[0]; p.w = (const uint8_t*)d_in[1];
  p.sx = (const float*)d_in[2]; p.sw = (const float*)d_in[3];
  p.out = (float*)d_out;
  for (int i = 0; i < NR; i++) p.ws[i] = (uint8_t*)d_ws_all[i];
  p.rank = rank; p.pad = 0;
  gemm_ag_kernel<<<NWG, NTHR, LDS_BYTES, stream>>>(p);
  HIP_CHECK(hipGetLastError());
}
```
